```python
import numpy as np

kernel_name = "gpu_gemm_ar_m4096_k4096_n16384_e5m2e5m2_dequant"


ABSMAX_RTOL = 0.05

COMPUTE_DTYPE = "float32"
EPILOGUE = "dequant"
IN_QUANT = None
OUT_QUANT = None
IN_FP8 = "e5m2e5m2"
IN_DTYPES = {"x": "e5m2", "w_mat": "e5m2",
             "scale_x": "float32", "scale_w": "float32"}
WORLD_SIZE = 8

def _snap_fp8(x, mant_bits, max_abs, min_normal):
    a = np.abs(x.astype(np.float64))
    _m, _e = np.frexp(a)
    q = np.where(a >= min_normal, np.ldexp(1.0, _e - 1 - mant_bits),
                 min_normal * 2.0 ** -mant_bits)
    snapped = np.minimum(np.round(a / q) * q, max_abs)
    return (np.sign(x) * snapped).astype(np.float32)

def setup_inputs():

    x = _snap_fp8(np.random.randn(4096, 4096).astype(np.float32), 2, 57344.0, 2.0 ** -14)
    w_mat = _snap_fp8(np.random.randn(4096, 16384).astype(np.float32), 2, 57344.0, 2.0 ** -14)
    scale_x = (0.02 * (1.0 + np.random.rand(1))).astype(np.float32)
    scale_w = (0.02 * (1.0 + np.random.rand(1))).astype(np.float32)
    return {"x": x, "w_mat": w_mat, "scale_x": scale_x, "scale_w": scale_w}

def reference(x, w_mat, scale_x, scale_w):
    acc = (x @ w_mat).astype(np.float32)
    y = (acc * (scale_x * scale_w)).astype(np.float32)
    return y
```

```cpp
#include <hip/hip_runtime.h>
#include <rccl/rccl.h>
#include <cstdio>
#include <utility>

#define HIP_CHECK(expr) do { hipError_t _e = (expr); if (_e != hipSuccess) { \
    fprintf(stderr, "HIP error %s at %s:%d\n", hipGetErrorString(_e), __FILE__, __LINE__); \
    return; } } while (0)

namespace {

typedef int i32x8 __attribute__((ext_vector_type(8)));
typedef int i32x4 __attribute__((ext_vector_type(4)));
typedef unsigned u32x4 __attribute__((ext_vector_type(4)));
typedef float f32x4 __attribute__((ext_vector_type(4)));
typedef __amdgpu_buffer_rsrc_t rsrc_t;
#define LAS __attribute__((address_space(3)))
#define DI __device__ __forceinline__
#define GAP() __builtin_amdgcn_sched_barrier(0)

constexpr int M = 4096, N = 16384, K = 4096, NR = 8, KL = K / NR;
constexpr unsigned XBYTES = (unsigned)M * K, WBYTES = (unsigned)N * K, STAGE_BYTES = XBYTES + WBYTES;
constexpr unsigned BLK = 32768, GRP = 16384, PANEL = 32 * BLK;
constexpr int GRID = 256, NPUSH = 16;
constexpr size_t WS_DONE = 0, WS_TMO = 128, WS_FLAGS = 4096, WS_STAGE = 1u << 20;
constexpr unsigned FLAG_BYTES = 80 * 32 * 8 * 4;
constexpr size_t WS_NEED = WS_STAGE + 2 * (size_t)STAGE_BYTES;
constexpr unsigned LDS_RING = 8 * GRP, LDS_BYTES = LDS_RING + 64;
constexpr unsigned long long DEADLINE = 40000000ull;
constexpr int AUX_SYS = 17, AUX_NT = 2;

struct Args {
  const unsigned char* x; const unsigned char* w; const float* sx; const float* sw; float* out;
  char* ws[NR]; int rank;
  unsigned long long* dbg;
};

constexpr int WAITCNT(int vm, int lgkm) { return (vm & 15) | (7 << 4) | ((lgkm & 15) << 8) | ((vm >> 4) << 14); }
DI rsrc_t mkrsrc(const void* p, unsigned bytes) { return __builtin_amdgcn_make_buffer_rsrc((void*)p, 0, bytes, 0x00020000); }

struct Ctx {
  i32x8 X[2][4], W[2][4];
  LAS char* lds;
  unsigned rdx, rdw, rdx2, rdw2;
  unsigned voff, m0w;
  rsrc_t rs;
  rsrc_t rf; unsigned fvoff0, fvoff1; unsigned call; unsigned* tmo; int ready; int wave; int rank;
};

constexpr int AI(int gx, int fx, int gw, int iw) { return (((gx * 4 + fx) * 2 + gw) * 4 + iw) * 4; }
template <int A, bool ZERO> DI void mfma(const i32x8& x, const i32x8& w) {
  if constexpr (ZERO) asm volatile("v_mfma_f32_16x16x128_f8f6f4 a[%c2:%c3], %0, %1, 0 cbsz:1 blgp:1" :: "v"(x), "v"(w), "i"(A), "i"(A + 3));
  else asm volatile("v_mfma_f32_16x16x128_f8f6f4 a[%c2:%c3], %0, %1, a[%c2:%c3] cbsz:1 blgp:1" :: "v"(x), "v"(w), "i"(A), "i"(A + 3));
}
template <int A> DI float accrd() { float r; asm volatile("v_accvgpr_read_b32 %0, a[%c1]" : "=v"(r) : "i"(A)); return r; }
#define A10(n) "a" #n "0", "a" #n "1", "a" #n "2", "a" #n "3", "a" #n "4", "a" #n "5", "a" #n "6", "a" #n "7", "a" #n "8", "a" #n "9"
#define ACLOB "a0","a1","a2","a3","a4","a5","a6","a7","a8","a9", A10(1),A10(2),A10(3),A10(4),A10(5),A10(6),A10(7),A10(8),A10(9),A10(10),A10(11),A10(12),A10(13),A10(14),A10(15),A10(16),A10(17),A10(18),A10(19),A10(20),A10(21),A10(22),A10(23),A10(24),"a250","a251","a252","a253","a254","a255"

template <int OFF, int I> DI void ldsread(const Ctx& c, i32x8 (&dst)[4], unsigned rd) {
  constexpr int f = I >> 1, h = I & 1;
  const i32x4 v = *(const LAS i32x4*)(c.lds + rd + OFF + f * 2048 + h * 1024);
  if constexpr (h == 0) { dst[f][0] = v[0]; dst[f][1] = v[1]; dst[f][2] = v[2]; dst[f][3] = v[3]; }
  else { dst[f][4] = v[0]; dst[f][5] = v[1]; dst[f][6] = v[2]; dst[f][7] = v[3]; }
}
template <int OFF, int... Is> DI void ldsread8_(const Ctx& c, i32x8 (&dst)[4], unsigned rd, std::integer_sequence<int, Is...>) { (ldsread<OFF, Is>(c, dst, rd), ...); }
template <int OFF> DI void ldsread8(const Ctx& c, i32x8 (&dst)[4], unsigned rd) { ldsread8_<OFF>(c, dst, rd, std::make_integer_sequence<int, 8>{}); }
template <int LOFF, int I> DI void dma(const Ctx& c, unsigned soff) {
  __builtin_amdgcn_raw_ptr_buffer_load_lds(c.rs, (LAS void*)(c.lds + LOFF + c.m0w), 16, c.voff, soff, I * 1024, 0);
}
template <int LOFF> DI void dma4(const Ctx& c, unsigned soff) { dma<LOFF, 0>(c, soff); dma<LOFF, 1>(c, soff); dma<LOFF, 2>(c, soff); dma<LOFF, 3>(c, soff); }

DI unsigned kstep_off(int rank, int o) { return (unsigned)((((rank + o) & 7) << 2) + (o >> 3)) * BLK; }

DI int poll_tile(const Ctx& c, int xp, int wp) {
  const int lane = threadIdx.x & 63;
  const unsigned pan = (lane & 2) ? (16u + wp) : (unsigned)xp;
  const u32x4 a = __builtin_amdgcn_raw_buffer_load_b128(c.rf, c.fvoff0 + pan * 1024u, 0, AUX_SYS);
  const u32x4 b = __builtin_amdgcn_raw_buffer_load_b128(c.rf, c.fvoff1 + pan * 1024u, 0, AUX_SYS);
  const unsigned cl = c.call;
  const bool oka = (a.x >= cl) & (a.y >= cl) & (a.z >= cl) & (a.w >= cl), okb = (b.x >= cl) & (b.y >= cl) & (b.z >= cl) & (b.w >= cl);
  const unsigned long long na = ~__ballot(oka), nb = ~__ballot(okb);
  if (na) return __builtin_ctzll(na) >> 2;
  if (nb) return 16 + (__builtin_ctzll(nb) >> 2);
  return 32;
}
DI void gate_slow(Ctx& c, int S, int xp, int wp) {
  const unsigned long long t0 = __builtin_amdgcn_s_memrealtime();
  const int base = S & ~31;
  for (;;) {
    c.ready = base + poll_tile(c, xp, wp);
    if (c.ready > S) break;
    if (__builtin_amdgcn_readfirstlane(__hip_atomic_load(c.tmo, __ATOMIC_RELAXED, __HIP_MEMORY_SCOPE_AGENT)) == c.call || __builtin_amdgcn_s_memrealtime() - t0 > DEADLINE) {
      __hip_atomic_store(c.tmo, c.call, __ATOMIC_RELAXED, __HIP_MEMORY_SCOPE_AGENT);
      *(volatile LAS unsigned*)(c.lds + LDS_RING) = 1u;
      c.ready = 1 << 20;
      break;
    }
    __builtin_amdgcn_s_sleep(4);
  }
}
DI void gate(Ctx& c, int S, int xp, int wp) {
  if (c.wave == 3 && __builtin_expect(S >= c.ready, 0)) gate_slow(c, S, xp, wp);
}

struct Gate { int S, xp, wp; };
template <int GX, int GW, bool ZERO, bool RISX, int RG, int RSLOT, int DSLOT, bool GATE, int Mi> DI void gap(Ctx& c, unsigned soff, const Gate& g) {
  constexpr int fx = Mi >> 2, iw = (fx & 1) ? 3 - (Mi & 3) : (Mi & 3);
  mfma<AI(GX, fx, GW, iw), ZERO>(c.X[GX][fx], c.W[GW][iw]);
  if constexpr (Mi < 8) { if constexpr (RISX) ldsread<(RSLOT & 3) * GRP, Mi>(c, c.X[RG], RSLOT < 4 ? c.rdx : c.rdx2); else ldsread<(RSLOT & 3) * GRP, Mi>(c, c.W[RG], RSLOT < 4 ? c.rdw : c.rdw2); }
  if constexpr (Mi >= 8 && Mi < 12) dma<DSLOT * GRP, Mi - 8>(c, soff);
  if constexpr (Mi == 15) {
    if constexpr (GATE) { GAP(); gate(c, g.S, g.xp, g.wp); GAP(); }
    __builtin_amdgcn_s_waitcnt(WAITCNT(24, 0));
    __builtin_amdgcn_s_barrier();
  }
  GAP();
}
template <int GX, int GW, bool ZERO, bool RISX, int RG, int RSLOT, int DSLOT, bool GATE, int... Ms> DI void quad_(Ctx& c, unsigned soff, const Gate& g, std::integer_sequence<int, Ms...>) {
  (gap<GX, GW, ZERO, RISX, RG, RSLOT, DSLOT, GATE, Ms>(c, soff, g), ...);
}
template <int GX, int GW, bool ZERO, bool RISX, int RG, int RSLOT, int DSLOT, bool GATE> DI void quad(Ctx& c, unsigned soff, const Gate& g) {
  quad_<GX, GW, ZERO, RISX, RG, RSLOT, DSLOT, GATE>(c, soff, g, std::make_integer_sequence<int, 16>{});
}
template <bool FIRST> DI void two_steps(Ctx& c, unsigned x2, unsigned w2, unsigned x3, unsigned w3, unsigned x4, const Gate& g3, const Gate& g4) {
  quad<0, 0, FIRST, false, 1, 2, 1, false>(c, w2, g3);
  quad<0, 1, FIRST, true, 1, 3, 2, false>(c, w2 + GRP, g3);
  quad<1, 1, FIRST, true, 0, 4, 3, true>(c, x2 + GRP, g3);
  quad<1, 0, FIRST, false, 1, 5, 4, false>(c, x3, g3);
  quad<0, 1, false, false, 0, 6, 5, false>(c, w3 + GRP, g4);
  quad<0, 0, false, true, 1, 7, 6, false>(c, w3, g4);
  quad<1, 0, false, true, 0, 0, 7, true>(c, x3 + GRP, g4);
  quad<1, 1, false, false, 0, 1, 0, false>(c, x4, g4);
}

template <int I> DI void epi_store(rsrc_t rc, unsigned vo, float scale) {
  constexpr int gw = I & 1, r = (I >> 1) & 3, fx = (I >> 3) & 3, gx = I >> 5;
  const f32x4 v = {accrd<AI(gx, fx, gw, 0) + r>() * scale, accrd<AI(gx, fx, gw, 1) + r>() * scale, accrd<AI(gx, fx, gw, 2) + r>() * scale, accrd<AI(gx, fx, gw, 3) + r>() * scale};
  __builtin_amdgcn_raw_buffer_store_b128(__builtin_bit_cast(u32x4, v), rc, vo + gw * 256, ((gx * 64 + fx * 16 + r) * N) * 4, AUX_NT);
  if constexpr (gw == 1) GAP();
}
template <int... Is> DI void epilogue_(rsrc_t rc, unsigned vo, float scale, std::integer_sequence<int, Is...>) { (epi_store<Is>(rc, vo, scale), ...); }
template <int I> DI void epi_store_q(rsrc_t rc, unsigned vo, float scale) {
  constexpr int r = I & 3, fx = I >> 2;
  const f32x4 v = {accrd<AI(0, fx, 0, 0) + r>() * scale, accrd<AI(0, fx, 0, 1) + r>() * scale, accrd<AI(0, fx, 0, 2) + r>() * scale, accrd<AI(0, fx, 0, 3) + r>() * scale};
  __builtin_amdgcn_raw_buffer_store_b128(__builtin_bit_cast(u32x4, v), rc, vo, ((fx * 16 + r) * N) * 4, AUX_NT);
  if constexpr (r & 1) GAP();
}
template <int... Is> DI void epilogue_q_(rsrc_t rc, unsigned vo, float scale, std::integer_sequence<int, Is...>) { (epi_store_q<Is>(rc, vo, scale), ...); }

template <int P, bool ZERO, int RS, int DS, int Mi> DI void gap_q(Ctx& c, unsigned so0, unsigned so1) {
  constexpr int fx = Mi >> 2, iw = (fx & 1) ? 3 - (Mi & 3) : (Mi & 3);
  mfma<AI(0, fx, 0, iw), ZERO>(c.X[P][fx], c.W[P][iw]);
  if constexpr (Mi < 8) ldsread<(RS & 1) * BLK, Mi>(c, c.X[P ^ 1], RS < 2 ? c.rdx : c.rdx2);
  else ldsread<(RS & 1) * BLK, Mi - 8>(c, c.W[P ^ 1], RS < 2 ? c.rdw : c.rdw2);
  if constexpr (Mi < 4) dma<DS * BLK, Mi>(c, so0);
  else if constexpr (Mi < 8) dma<DS * BLK + 4096, Mi - 4>(c, so1);
  if constexpr (Mi == 15) { __builtin_amdgcn_s_waitcnt(WAITCNT(16, 0)); __builtin_amdgcn_s_barrier(); }
  GAP();
}
template <int P, bool ZERO, int RS, int DS, int... Ms> DI void step_q_(Ctx& c, unsigned so, std::integer_sequence<int, Ms...>) { (gap_q<P, ZERO, RS, DS, Ms>(c, so, so + 4096), ...); }
template <int P, bool ZERO, int RS, int DS> DI void step_q(Ctx& c, unsigned so) { step_q_<P, ZERO, RS, DS>(c, so, std::make_integer_sequence<int, 16>{}); }

DI void item_of(int i, bool& isx, int& p, int& j) {
  if (i < 124) { j = i / 31; const int t = i - j * 31; isx = !(t & 1); p = t >> 1; }
  else if (i < 304) { const int q = i - 124, r = q / 60, q2 = q - r * 60; j = q2 / 15; p = 15 * (r + 1) + (q2 - j * 15); isx = false; }
  else { const int q = i - 304; j = q >> 2; p = 60 + (q & 3); isx = false; }
}
struct Unit { bool isx; unsigned src, dst, flag; };
DI Unit unit_of(int u, int rank) {
  Unit r; int p, j; item_of(u >> 3, r.isx, p, j);
  const int sub = u & 7, w = sub >> 2, g = (sub >> 1) & 1, h = sub & 1, ks = rank * 4 + j;
  r.src = r.isx ? (unsigned)((p * 256 + w * 128 + g * 64) * KL + j * 128 + h * 64) : (unsigned)((j * 128 + h * 64) * N + p * 256 + w * 128 + g * 64);
  r.dst = (r.isx ? 0u : XBYTES) + (unsigned)((p * 32 + ks) * 2 + g) * GRP + w * 8192 + h * 1024;
  r.flag = (unsigned)((((r.isx ? p : 16 + p) * 32 + ks) * 8 + sub) * 4);
  return r;
}
DI void load_raw(unsigned (&raw)[16], const Unit& u, rsrc_t rx, rsrc_t rw, unsigned vx, unsigned vw) {
  if (u.isx) {
#pragma unroll
    for (int f = 0; f < 4; ++f) { const u32x4 v = __builtin_amdgcn_raw_buffer_load_b128(rx, vx, u.src + f * 16 * KL, 0); raw[4 * f] = v.x; raw[4 * f + 1] = v.y; raw[4 * f + 2] = v.z; raw[4 * f + 3] = v.w; }
  } else {
#pragma unroll
    for (int b = 0; b < 16; ++b) raw[b] = __builtin_amdgcn_raw_buffer_load_b32(rw, vw, u.src + b * N, 0);
  }
}
DI void cook(u32x4 (&out)[4], const unsigned (&raw)[16], bool isx) {
  if (isx) {
#pragma unroll
    for (int f = 0; f < 4; ++f) out[f] = (u32x4){raw[4 * f], raw[4 * f + 1], raw[4 * f + 2], raw[4 * f + 3]};
  } else {
#pragma unroll
    for (int b = 0; b < 4; ++b) {
      const unsigned r0 = raw[4 * b], r1 = raw[4 * b + 1], r2 = raw[4 * b + 2], r3 = raw[4 * b + 3];
      const unsigned t0 = __builtin_amdgcn_perm(r1, r0, 0x05010400u), t1 = __builtin_amdgcn_perm(r1, r0, 0x07030602u);
      const unsigned t2 = __builtin_amdgcn_perm(r3, r2, 0x05010400u), t3 = __builtin_amdgcn_perm(r3, r2, 0x07030602u);
      out[0][b] = __builtin_amdgcn_perm(t2, t0, 0x05040100u); out[1][b] = __builtin_amdgcn_perm(t2, t0, 0x07060302u);
      out[2][b] = __builtin_amdgcn_perm(t3, t1, 0x05040100u); out[3][b] = __builtin_amdgcn_perm(t3, t1, 0x07060302u);
    }
  }
}
DI void pusher(const Args& a, unsigned call, int wave, int lane) {
  constexpr int NW = NPUSH * 4, NU = 2560;
  const int gw = blockIdx.x * 4 + wave;
  const rsrc_t rx = mkrsrc(a.x, M * KL), rw = mkrsrc(a.w, KL * N);
  rsrc_t rd[NR];
  char* fl = nullptr;
#pragma unroll
  for (int i = 0; i < NR; ++i) {
    const int d = (i + gw) & 7;
    char* p = a.ws[0];
#pragma unroll
    for (int q = 1; q < NR; ++q) p = d == q ? a.ws[q] : p;
    rd[i] = mkrsrc(p + WS_STAGE + (size_t)(call & 1) * STAGE_BYTES, STAGE_BYTES);
    fl = (lane & 7) == i ? a.ws[i] + WS_FLAGS : fl;
  }
  const int cc = lane & 15, q = lane >> 4;
  const unsigned vx = cc * KL + q * 16, vw = q * 16 * N + cc * 4, vd = lane * 16;
  unsigned raw[16]; u32x4 out[4];
  Unit cur = unit_of(gw, a.rank);
  load_raw(raw, cur, rx, rw, vx, vw);
  unsigned prev_flag = 0; bool have_prev = false;
  for (int u = gw; u < NU; u += NW) {
    cook(out, raw, cur.isx);
    const Unit nxt = unit_of(u + NW < NU ? u + NW : u, a.rank);
    load_raw(raw, nxt, rx, rw, vx, vw);
    GAP();
#pragma unroll
    for (int s = 0; s < 12; ++s) __builtin_amdgcn_raw_buffer_store_b128(out[s >> 3], rd[s & 7], vd + (s >> 3) * 2048, cur.dst, AUX_SYS);
    asm volatile("s_waitcnt vmcnt(12)" ::: "memory");
    if (have_prev && lane < 8) __hip_atomic_store((unsigned*)(fl + prev_flag), call, __ATOMIC_RELAXED, __HIP_MEMORY_SCOPE_SYSTEM);
    GAP();
#pragma unroll
    for (int s = 12; s < 32; ++s) __builtin_amdgcn_raw_buffer_store_b128(out[s >> 3], rd[s & 7], vd + (s >> 3) * 2048, cur.dst, AUX_SYS);
    GAP();
    prev_flag = cur.flag; have_prev = true; cur = nxt;
  }
  asm volatile("s_waitcnt vmcnt(0)" ::: "memory");
  if (lane < 8) __hip_atomic_store((unsigned*)(fl + prev_flag), call, __ATOMIC_RELAXED, __HIP_MEMORY_SCOPE_SYSTEM);
}

DI void tile_of(int round, int b, int& mt, int& nt) {
  const int x = b & 7, idx = (b >> 3) - NPUSH / 8, p = (4 * round + (x & 3)) * 30 + idx;
  nt = p >> 3; mt = (x >> 2) * 8 + (p & 7);
}

#define STAMP(i)

__global__ void __launch_bounds__(256, 1) fused_gemm_gather(Args a) {
  extern __shared__ __attribute__((aligned(16))) char smem[];
  const int tid = threadIdx.x, lane = tid & 63, wave = __builtin_amdgcn_readfirstlane(tid >> 6);
  char* myws = a.ws[0];
#pragma unroll
  for (int q = 1; q < NR; ++q) myws = a.rank == q ? a.ws[q] : myws;
  STAMP(0);
  const unsigned call = __builtin_amdgcn_readfirstlane(__hip_atomic_load((unsigned*)(myws + WS_DONE), __ATOMIC_RELAXED, __HIP_MEMORY_SCOPE_AGENT)) / GRID + 1;
  float scale = a.sx[0] * a.sw[0];
  Ctx c;
  const int wr = wave >> 1, wc = wave & 1, fr = lane & 15, fq = lane >> 4;
  c.lds = (LAS char*)smem;
  c.rs = mkrsrc(myws + WS_STAGE + (size_t)(call & 1) * STAGE_BYTES, STAGE_BYTES);
  c.rf = mkrsrc(myws + WS_FLAGS, FLAG_BYTES);
  { const int o = lane >> 2; c.fvoff0 = kstep_off(a.rank, o) / BLK * 32 + (lane & 1) * 16; c.fvoff1 = kstep_off(a.rank, o + 16) / BLK * 32 + (lane & 1) * 16; }
  c.call = call; c.tmo = (unsigned*)(myws + WS_TMO); c.ready = 0; c.wave = wave; c.rank = a.rank;
  if (tid == 0) *(volatile LAS unsigned*)(c.lds + LDS_RING) = 0u;
  const unsigned vo = (fq * 4 * N + fr * 4) * 4;
  asm volatile("" ::: ACLOB);

  if (blockIdx.x < NPUSH) {
    pusher(a, call, wave, lane);
    STAMP(1);
  } else {
    c.rdx = wr * 8192 + lane * 16; c.rdw = wc * 8192 + lane * 16; c.rdx2 = c.rdx + 65536; c.rdw2 = c.rdw + 65536;
    asm volatile("" : "+v"(c.rdx2), "+v"(c.rdw2));
    c.voff = wave * 4096 + lane * 16; c.m0w = wave * 4096;
    int mt, nt; tile_of(0, blockIdx.x, mt, nt);
    unsigned xb = mt * PANEL, wb = XBYTES + nt * PANEL;
    __builtin_amdgcn_s_waitcnt(WAITCNT(63, 0));
    gate(c, 2, mt, nt);
    __builtin_amdgcn_s_barrier();
    { const unsigned k0 = kstep_off(a.rank, 0), k1 = kstep_off(a.rank, 1), k2 = kstep_off(a.rank, 2);
      dma4<0 * GRP>(c, xb + k0); dma4<1 * GRP>(c, wb + k0); dma4<2 * GRP>(c, wb + k0 + GRP); dma4<3 * GRP>(c, xb + k0 + GRP);
      dma4<4 * GRP>(c, xb + k1); dma4<5 * GRP>(c, wb + k1 + GRP); dma4<6 * GRP>(c, wb + k1); dma4<7 * GRP>(c, xb + k1 + GRP);
      __builtin_amdgcn_s_waitcnt(WAITCNT(24, 15));
      __builtin_amdgcn_s_barrier();
      ldsread8<0 * GRP>(c, c.X[0], c.rdx); ldsread8<1 * GRP>(c, c.W[0], c.rdw);
      __builtin_amdgcn_s_waitcnt(WAITCNT(20, 0));
      __builtin_amdgcn_s_barrier();
      dma4<0 * GRP>(c, xb + k2); }
    GAP();
    STAMP(1);
#pragma unroll 1
    for (int round = 0; round < 4; ++round) {
      int mt2, nt2; tile_of(round + 1, blockIdx.x, mt2, nt2);
      const bool more = round < 3;
      const unsigned xn = more ? mt2 * PANEL : 0xC0000000u, wn = more ? XBYTES + nt2 * PANEL : 0xC0000000u;
      const int S0 = round * 32;
      two_steps<true>(c, xb + kstep_off(a.rank, 2), wb + kstep_off(a.rank, 2), xb + kstep_off(a.rank, 3), wb + kstep_off(a.rank, 3), xb + kstep_off(a.rank, 4), Gate{S0 + 3, mt, nt}, Gate{S0 + 4, mt, nt});
#pragma unroll 1
      for (int k = 2; k < 28; k += 2) {
        const unsigned k2 = kstep_off(a.rank, k + 2), k3 = kstep_off(a.rank, k + 3), k4 = kstep_off(a.rank, k + 4);
        two_steps<false>(c, xb + k2, wb + k2, xb + k3, wb + k3, xb + k4, Gate{S0 + k + 3, mt, nt}, Gate{S0 + k + 4, mt, nt});
      }
      { const unsigned k30 = kstep_off(a.rank, 30), k31 = kstep_off(a.rank, 31), k0 = kstep_off(a.rank, 0), k1 = kstep_off(a.rank, 1), k2 = kstep_off(a.rank, 2);
        const int big = more ? 0 : -(1 << 20);
        two_steps<false>(c, xb + k30, wb + k30, xb + k31, wb + k31, xn + k0, Gate{S0 + 31, mt, nt}, Gate{S0 + 32 + big, mt2, nt2});
        two_steps<false>(c, xn + k0, wn + k0, xn + k1, wn + k1, xn + k2, Gate{S0 + 33 + big, mt2, nt2}, Gate{S0 + 34 + big, mt2, nt2}); }
      STAMP(2 + 2 * round);
      const rsrc_t rc = mkrsrc((char*)a.out + ((size_t)(mt * 256 + wr * 128) * N + nt * 256 + wc * 128) * 4, 128u * N * 4);
      const float sc = *(volatile LAS unsigned*)(c.lds + LDS_RING) ? __builtin_nanf("") : scale;
      asm volatile("s_nop 15" ::: ACLOB);
      epilogue_(rc, vo, sc, std::make_integer_sequence<int, 64>{});
      GAP();
      STAMP(3 + 2 * round);
      mt = mt2; nt = nt2; xb = xn; wb = wn;
    }
  }
  {
    const int x = blockIdx.x & 7, idx = blockIdx.x >> 3;
    const int nt = 60 + (x & 3), mt = (x >> 2) * 8 + (idx >> 2), hx = (idx >> 1) & 1, hw = idx & 1;
    c.rdx = wr * 8192 + lane * 16; c.rdw = 16384 + wc * 8192 + lane * 16; c.rdx2 = c.rdx + 65536; c.rdw2 = c.rdw + 65536;
    asm volatile("" : "+v"(c.rdx2), "+v"(c.rdw2));
    c.voff = lane * 16; c.m0w = wave * 8192;
    const unsigned sb = (wave < 2 ? mt * PANEL + hx * 8192 : XBYTES + nt * PANEL + hw * 8192) + (wave & 1) * GRP;
    __builtin_amdgcn_s_waitcnt(WAITCNT(0, 0));
    c.ready = 0;
    gate(c, 31, mt, nt);
    __builtin_amdgcn_s_barrier();
    STAMP(10);
#define DMA8(S, o) do { dma4<(S) * BLK>(c, sb + (o) * BLK); dma4<(S) * BLK + 4096>(c, sb + (o) * BLK + 4096); } while (0)
    DMA8(0, 0); DMA8(1, 1); DMA8(2, 2); DMA8(3, 3);
    __builtin_amdgcn_s_waitcnt(WAITCNT(24, 15));
    __builtin_amdgcn_s_barrier();
    ldsread8<0>(c, c.X[0], c.rdx); ldsread8<0>(c, c.W[0], c.rdw);
    __builtin_amdgcn_s_waitcnt(WAITCNT(16, 0));
    __builtin_amdgcn_s_barrier();
    GAP();
    step_q<0, true, 1, 0>(c, sb + 4 * BLK); step_q<1, false, 2, 1>(c, sb + 5 * BLK); step_q<0, false, 3, 2>(c, sb + 6 * BLK); step_q<1, false, 0, 3>(c, sb + 7 * BLK);
#pragma unroll 1
    for (int k = 4; k < 32; k += 4) {
      const unsigned so = k < 28 ? sb + (k + 4) * BLK : 0xC0000000u;
      step_q<0, false, 1, 0>(c, so); step_q<1, false, 2, 1>(c, so + BLK); step_q<0, false, 3, 2>(c, so + 2 * BLK); step_q<1, false, 0, 3>(c, so + 3 * BLK);
    }
    STAMP(11);
    const rsrc_t rc = mkrsrc((char*)a.out + ((size_t)(mt * 256 + hx * 128 + wr * 64) * N + nt * 256 + hw * 128 + wc * 64) * 4, 64u * N * 4);
    const float sc = *(volatile LAS unsigned*)(c.lds + LDS_RING) ? __builtin_nanf("") : scale;
    asm volatile("s_nop 15" ::: ACLOB);
    epilogue_q_(rc, vo, sc, std::make_integer_sequence<int, 16>{});
  }
  if (tid == 0) __hip_atomic_fetch_add((unsigned*)(myws + WS_DONE), 1u, __ATOMIC_RELAXED, __HIP_MEMORY_SCOPE_AGENT);
}

}

extern "C" void kernel_launch(int rank, int nranks, ncclComm_t comm, hipStream_t stream,
                              void* const* d_in, const int* in_sizes, int n_in,
                              void* d_out, int out_size, const int* in_dtypes, int out_dtype,
                              void* const* d_out_all, void* const* d_ws_all, size_t ws_bytes) {
  if (nranks != NR || (unsigned)rank >= (unsigned)NR || n_in != 4 || in_sizes[0] != M * KL || in_sizes[1] != KL * N || out_size != M * N || ws_bytes < WS_NEED) {
    fprintf(stderr, "kernel_launch: unexpected problem shape\n");
    return;
  }
  Args a;
  a.x = (const unsigned char*)d_in[0]; a.w = (const unsigned char*)d_in[1]; a.sx = (const float*)d_in[2]; a.sw = (const float*)d_in[3];
  a.out = (float*)d_out; a.rank = rank; a.dbg = nullptr;
  for (int p = 0; p < NR; ++p) a.ws[p] = (char*)d_ws_all[p];
  HIP_CHECK(hipFuncSetAttribute((const void*)fused_gemm_gather, hipFuncAttributeMaxDynamicSharedMemorySize, LDS_BYTES));
  hipLaunchKernelGGL(fused_gemm_gather, dim3(GRID), dim3(256), LDS_BYTES, stream, a);
  HIP_CHECK(hipGetLastError());
}
```

```cpp
#include <hip/hip_runtime.h>
#include <rccl/rccl.h>
#include <cstdio>
#include <utility>
#pragma clang diagnostic ignored "-Winline-asm"

#define HIP_CHECK(expr) do { hipError_t _e = (expr); if (_e != hipSuccess) { \
    fprintf(stderr, "HIP error %s at %s:%d\n", hipGetErrorString(_e), __FILE__, __LINE__); \
    return; } } while (0)

namespace {

typedef int i32x8 __attribute__((ext_vector_type(8)));
typedef int i32x4 __attribute__((ext_vector_type(4)));
typedef unsigned u32x4 __attribute__((ext_vector_type(4)));
typedef float f32x4 __attribute__((ext_vector_type(4)));
typedef __amdgpu_buffer_rsrc_t rsrc_t;
#define LAS __attribute__((address_space(3)))
#define DI __device__ __forceinline__
#define GAP() __builtin_amdgcn_sched_barrier(0)

constexpr int M = 4096, N = 16384, K = 4096, NR = 8, KL = K / NR;
constexpr unsigned XBYTES = (unsigned)M * K, WBYTES = (unsigned)N * K, STAGE_BYTES = XBYTES + WBYTES;
constexpr unsigned BLK = 32768, GRP = 16384, PANEL = 32 * BLK;
#define POLL_AUX 17
#define POLL_SLEEP 2
#define PWAIT 12
#define NPUSH_ 16
#define NFULL_ 128
constexpr int GRID = 256, NPUSH = NPUSH_, GPX = (GRID - NPUSH) / 8;
#define KPHASE 2200
#define KCLK 546
constexpr int NFULL = NFULL_, NITEM = NFULL + (128 - NFULL) * 4 + 32 - GPX;
constexpr size_t WS_DONE = 0, WS_TMO = 128, WS_QUEUE = 256, WS_UNITQ = 2048, WS_FLAGS = 4096, WS_STAGE = 1u << 20;
constexpr unsigned FLAG_BYTES = 80 * 32 * 8 * 4;
constexpr size_t WS_NEED = WS_STAGE + 2 * (size_t)STAGE_BYTES;
constexpr unsigned LDS_RING = 8 * GRP, LDS_DEAD = LDS_RING, LDS_TICKET = LDS_RING + 4, LDS_SEL = LDS_RING + 16, LDS_SNAP = LDS_RING + 64, LDS_BYTES = LDS_SNAP + 8 * 2048;
constexpr unsigned long long DEADLINE = 40000000ull;
#define OUT_AUX 2
constexpr int AUX_SYS = 17, AUX_NT = OUT_AUX;
constexpr unsigned OOB = 0x40000000u;

struct Args {
  const unsigned char* x; const unsigned char* w; const float* sx; const float* sw; float* out;
  char* ws[NR]; int rank;
  int kphase, kclk, useclk;
  unsigned order[160];
  unsigned long long* dbg;
};

constexpr int WAITCNT(int vm, int lgkm) { return (vm & 15) | (7 << 4) | ((lgkm & 15) << 8) | ((vm >> 4) << 14); }
DI rsrc_t mkrsrc(const void* p, unsigned bytes) {
  const unsigned long long v = (unsigned long long)p;
  const unsigned long long u = ((unsigned long long)(unsigned)__builtin_amdgcn_readfirstlane((unsigned)(v >> 32)) << 32) | (unsigned)__builtin_amdgcn_readfirstlane((unsigned)v);
  return __builtin_amdgcn_make_buffer_rsrc((void*)u, 0, bytes, 0x00020000);
}

struct Ctx {
  i32x8 X[2][4], W[2][4];
  LAS char* lds;
  unsigned rdx, rdw, rdx2, rdw2;
  unsigned voff, m0w;
  rsrc_t rs;
  unsigned x2, w2, x3, w3;
  rsrc_t rf, rq; unsigned fv; unsigned call; unsigned* tmo; int wave;
  unsigned av, kn, ks, ptr, kclk; int quota, ti, snap, pos, npos, qmode; unsigned px, pw;
  unsigned trc, trc2; unsigned long long tbase;
};

constexpr int AI(int gx, int fx, int gw, int iw) { return (((gx * 4 + fx) * 2 + gw) * 4 + iw) * 4; }
template <int A, bool ZERO> DI void mfma(const i32x8& x, const i32x8& w) {
  if constexpr (ZERO) asm volatile("v_mfma_f32_16x16x128_f8f6f4 a[%c2:%c3], %0, %1, 0 cbsz:1 blgp:1" :: "v"(x), "v"(w), "i"(A), "i"(A + 3));
  else asm volatile("v_mfma_f32_16x16x128_f8f6f4 a[%c2:%c3], %0, %1, a[%c2:%c3] cbsz:1 blgp:1" :: "v"(x), "v"(w), "i"(A), "i"(A + 3));
}
template <int A> DI float accrd() { float r; asm volatile("v_accvgpr_read_b32 %0, a[%c1]" : "=v"(r) : "i"(A)); return r; }
#define A10(n) "a" #n "0", "a" #n "1", "a" #n "2", "a" #n "3", "a" #n "4", "a" #n "5", "a" #n "6", "a" #n "7", "a" #n "8", "a" #n "9"
#define ACLOB "a0","a1","a2","a3","a4","a5","a6","a7","a8","a9", A10(1),A10(2),A10(3),A10(4),A10(5),A10(6),A10(7),A10(8),A10(9),A10(10),A10(11),A10(12),A10(13),A10(14),A10(15),A10(16),A10(17),A10(18),A10(19),A10(20),A10(21),A10(22),A10(23),A10(24),"a250","a251","a252","a253","a254","a255"

template <int OFF, int I> DI void ldsread(const Ctx& c, i32x8 (&dst)[4], unsigned rd) {
  constexpr int f = I >> 1, h = I & 1;
  const i32x4 v = *(const LAS i32x4*)(c.lds + rd + OFF + f * 2048 + h * 1024);
  if constexpr (h == 0) { dst[f][0] = v[0]; dst[f][1] = v[1]; dst[f][2] = v[2]; dst[f][3] = v[3]; }
  else { dst[f][4] = v[0]; dst[f][5] = v[1]; dst[f][6] = v[2]; dst[f][7] = v[3]; }
}
template <int OFF, int... Is> DI void ldsread8_(const Ctx& c, i32x8 (&dst)[4], unsigned rd, std::integer_sequence<int, Is...>) { (ldsread<OFF, Is>(c, dst, rd), ...); }
template <int OFF> DI void ldsread8(const Ctx& c, i32x8 (&dst)[4], unsigned rd) { ldsread8_<OFF>(c, dst, rd, std::make_integer_sequence<int, 8>{}); }
template <int LOFF, int I> DI void dma(const Ctx& c, unsigned soff) {
  __builtin_amdgcn_raw_ptr_buffer_load_lds(c.rs, (LAS void*)(c.lds + LOFF + c.m0w), 16, c.voff, soff, I * 1024, 0);
}
template <int LOFF> DI void dma4(const Ctx& c, unsigned soff) { dma<LOFF, 0>(c, soff); dma<LOFF, 1>(c, soff); dma<LOFF, 2>(c, soff); dma<LOFF, 3>(c, soff); }

DI void tile_of(int x, int pos, int& mt, int& nt) { mt = (pos & 7) * 2 + (x >> 2); nt = (pos >> 3) * 4 + (x & 3); }
#define TRACE_S(c, S)

DI unsigned flag_voff(int xp, int wp) { const int lane = threadIdx.x & 63; return ((lane & 32) ? 16u + wp : (unsigned)xp) * 1024u + (lane & 31) * 32u; }
DI unsigned take_flags(const Ctx& c, const u32x4& a, const u32x4& b) {
  const unsigned cl = c.call;
  const unsigned long long m = __ballot((a.x >= cl) & (a.y >= cl) & (a.z >= cl) & (a.w >= cl) & (b.x >= cl) & (b.y >= cl) & (b.z >= cl) & (b.w >= cl));
  return (unsigned)m & (unsigned)(m >> 32);
}
DI void snap_issue(const Ctx& c, unsigned fv, int region) {
  __builtin_amdgcn_raw_ptr_buffer_load_lds(c.rf, (LAS void*)(c.lds + LDS_SNAP + region * 2048), 16, fv, 0, 0, POLL_AUX);
  __builtin_amdgcn_raw_ptr_buffer_load_lds(c.rf, (LAS void*)(c.lds + LDS_SNAP + region * 2048 + 1024), 16, fv, 16, 0, POLL_AUX);
}
DI unsigned snap_take(const Ctx& c, int region) {
  const unsigned o = LDS_SNAP + region * 2048 + (threadIdx.x & 63) * 16;
  const u32x4 a = *(volatile LAS u32x4*)(c.lds + o), b = *(volatile LAS u32x4*)(c.lds + o + 1024);
  return take_flags(c, a, b);
}
DI unsigned wait_flags(const Ctx& c, unsigned skip, bool all) {
  const unsigned long long t0 = __builtin_amdgcn_s_memrealtime();
  for (;;) {
    const u32x4 a = __builtin_amdgcn_raw_buffer_load_b128(c.rf, c.fv, 0, POLL_AUX), b = __builtin_amdgcn_raw_buffer_load_b128(c.rf, c.fv + 16, 0, POLL_AUX);
    const unsigned m = take_flags(c, a, b);
    if (all ? m == ~0u : (m & ~skip) != 0) return m;
    if (__builtin_amdgcn_readfirstlane(__hip_atomic_load(c.tmo, __ATOMIC_RELAXED, __HIP_MEMORY_SCOPE_AGENT)) == c.call || __builtin_amdgcn_s_memrealtime() - t0 > DEADLINE) {
      __hip_atomic_store(c.tmo, c.call, __ATOMIC_RELAXED, __HIP_MEMORY_SCOPE_AGENT);
      *(volatile LAS unsigned*)(c.lds + LDS_DEAD) = 1u;
      return ~0u;
    }
    __builtin_amdgcn_s_sleep(POLL_SLEEP);
  }
}
DI void open_tile(Ctx& c, int pos, int list = blockIdx.x & 7) {
  int mt, nt; tile_of(list, pos, mt, nt);
  c.pos = pos; c.kn = 0; c.av = 0; c.quota = 0;
  c.px = __builtin_amdgcn_readfirstlane(mt * PANEL); c.pw = __builtin_amdgcn_readfirstlane(XBYTES + nt * PANEL); c.fv = flag_voff(mt, nt);
}
DI void refresh(Ctx& c) {
  unsigned us = c.kn ^ c.av;
  const bool sw = us == ~0u;
  if (sw || c.snap || c.av == 0) {
    typedef unsigned u32x2 __attribute__((ext_vector_type(2)));
    if (c.wave == 3) {
      if (sw) { open_tile(c, c.npos); c.ti += 1; us = 0; c.snap = c.ti < 8; }
      unsigned m = c.kn;
      if (c.pos < NFULL || c.qmode) {
        if (c.snap) m |= snap_take(c, c.ti);
        if ((m & ~us) == 0) m = wait_flags(c, us, false);
      }
      const unsigned tp = (unsigned)((((__builtin_amdgcn_s_memrealtime() - c.tbase) & 0xffffffu) * c.kclk) >> 16) & 31u;
      *(volatile LAS u32x2*)(c.lds + LDS_SEL) = (u32x2){m, (unsigned)c.pos | (tp << 16)};
    }
    __builtin_amdgcn_s_waitcnt(WAITCNT(63, 0));
    __builtin_amdgcn_s_barrier();
    const u32x2 v = *(volatile LAS u32x2*)(c.lds + LDS_SEL);
    if (sw) {
      const int pos = __builtin_amdgcn_readfirstlane(v.y) & 0xffff;
      if (c.kclk) c.ptr = __builtin_amdgcn_readfirstlane(v.y) >> 16;
      if (c.wave != 3) { open_tile(c, pos); c.ti += 1; }
      us = 0;
      if (pos >= NFULL) { c.px = OOB; c.pw = OOB; c.kn = ~0u; c.av = ~0u; c.quota = 16; c.snap = 0; return; }
    }
    c.kn = __builtin_amdgcn_readfirstlane(v.x) | us;
    c.av = c.kn ^ us; c.snap = 0;
  }
  const int cnt = __builtin_popcount(us), have = __builtin_popcount(c.av);
  int q = (cnt < 24 ? 24 : cnt < 28 ? 28 : 32) - cnt;
  if (cnt + have < 32) { if (have <= 4) { if (c.wave == 3 && c.ti < 8) snap_issue(c, c.fv, c.ti); c.snap = c.ti < 8; q = 1; } else q = q < have - 4 ? q : have - 4; }
  if (c.wave == 3 && !c.qmode) {
    if (cnt == 24) {
      const unsigned qv = (threadIdx.x & 63) == 0 ? (blockIdx.x & 7) * 128u : OOB;
      asm volatile("v_mov_b32 v200, 1\n\ts_nop 4\n\tbuffer_atomic_add v200, %0, %1, 0 offen sc0" :: "v"(qv), "s"(c.rq) : "v200", "memory");
    }
    if (cnt == 28) {
      unsigned t;
      do { asm volatile("v_readfirstlane_b32 %0, v200" : "=s"(t)); } while (t == 1u);
      c.npos = t - c.call * 1024u;
      if (c.npos < NFULL && c.ti < 7) { int mt, nt; tile_of(blockIdx.x & 7, c.npos, mt, nt); snap_issue(c, flag_voff(mt, nt), c.ti + 1); }
    }
  }
  c.quota = q;
}
DI void pick0(Ctx& c) { if (__builtin_expect(c.quota == 0, 0)) refresh(c); TRACE_S(c, c.ti * 32 + __builtin_popcount(c.kn ^ c.av)); c.quota -= 1; asm volatile("" : "+s"(c.quota)); }
DI void pick1a(Ctx& c) { const unsigned long long d = (((unsigned long long)c.av << 32) | c.av) >> c.ptr; c.ks = (c.ptr + __builtin_ctz((unsigned)d)) & 31; asm volatile("" : "+s"(c.ks)); }
DI void pick1b(Ctx& c) { c.av &= ~(1u << c.ks); c.ptr = (c.ks + 1) & 31; c.ks *= BLK; asm volatile("" : "+s"(c.ks), "+s"(c.av), "+s"(c.ptr)); }
DI void pick1(Ctx& c) { pick1a(c); pick1b(c); }
DI void pick2(Ctx& c, unsigned& x, unsigned& w) { x = c.px + c.ks; w = c.pw + c.ks; asm volatile("" : "+s"(x), "+s"(w)); }

template <int GX, int GW, bool ZERO, bool RISX, int RG, int RSLOT, int DSLOT, int DSEL, int Mi> DI void gap(Ctx& c, unsigned& so) {
  constexpr int fx = Mi >> 2, iw = (fx & 1) ? 3 - (Mi & 3) : (Mi & 3);
  mfma<AI(GX, fx, GW, iw), ZERO>(c.X[GX][fx], c.W[GW][iw]);
  if constexpr (Mi < 8) { if constexpr (RISX) ldsread<(RSLOT & 3) * GRP, Mi>(c, c.X[RG], RSLOT < 4 ? c.rdx : c.rdx2); else ldsread<(RSLOT & 3) * GRP, Mi>(c, c.W[RG], RSLOT < 4 ? c.rdw : c.rdw2); }
  if constexpr (Mi == 5) { so = DSEL == 0 ? c.w2 : DSEL == 1 ? c.w2 + GRP : DSEL == 2 ? c.x2 + GRP : c.x2; asm volatile("" : "+s"(so)); }
  if constexpr (Mi >= 8 && Mi < 12) dma<DSLOT * GRP, Mi - 8>(c, so);
  if constexpr (DSEL == 2) { if constexpr (Mi == 12) pick0(c); if constexpr (Mi == 13) pick1a(c); if constexpr (Mi == 14) pick1b(c); }
  if constexpr (DSEL == 3 && Mi == 2) pick2(c, c.x2, c.w2);
  if constexpr (Mi == 15) {
    __builtin_amdgcn_s_waitcnt(WAITCNT(24, 0));
    __builtin_amdgcn_s_barrier();
  }
  GAP();
}
template <int GX, int GW, bool ZERO, bool RISX, int RG, int RSLOT, int DSLOT, int DSEL, int... Ms> DI void quad_(Ctx& c, std::integer_sequence<int, Ms...>) {
  unsigned so = 0;
  (gap<GX, GW, ZERO, RISX, RG, RSLOT, DSLOT, DSEL, Ms>(c, so), ...);
}
template <int GX, int GW, bool ZERO, bool RISX, int RG, int RSLOT, int DSLOT, int DSEL> DI void quad(Ctx& c) {
  quad_<GX, GW, ZERO, RISX, RG, RSLOT, DSLOT, DSEL>(c, std::make_integer_sequence<int, 16>{});
}
template <bool FIRST> DI void two_steps(Ctx& c) {
  quad<0, 0, FIRST, false, 1, 2, 1, 0>(c);
  quad<0, 1, FIRST, true, 1, 3, 2, 1>(c);
  quad<1, 1, FIRST, true, 0, 4, 3, 2>(c);
  quad<1, 0, FIRST, false, 1, 5, 4, 3>(c);
  quad<0, 1, false, false, 0, 6, 5, 1>(c);
  quad<0, 0, false, true, 1, 7, 6, 0>(c);
  quad<1, 0, false, true, 0, 0, 7, 2>(c);
  quad_<1, 1, false, false, 0, 1, 0, 3>(c, std::make_integer_sequence<int, 14>{});
}
DI void two_steps_tail(Ctx& c) { unsigned so = 0; gap<1, 1, false, false, 0, 1, 0, 3, 14>(c, so); gap<1, 1, false, false, 0, 1, 0, 3, 15>(c, so); }

template <int I> DI void epi_store(rsrc_t rc, unsigned vo, unsigned& so, float scale) {
  constexpr int gw = I & 1, r = (I >> 1) & 3, fx = (I >> 3) & 3, gx = I >> 5;
  const f32x4 v = {accrd<AI(gx, fx, gw, 0) + r>() * scale, accrd<AI(gx, fx, gw, 1) + r>() * scale, accrd<AI(gx, fx, gw, 2) + r>() * scale, accrd<AI(gx, fx, gw, 3) + r>() * scale};
  __builtin_amdgcn_raw_buffer_store_b128(__builtin_bit_cast(u32x4, v), rc, vo + gw * 256, so, AUX_NT);
  if constexpr (gw == 1) { so += (r == 3 ? 13 : 1) * N * 4; asm volatile("" : "+s"(so)); GAP(); }
}
template <int... Is> DI void epilogue_(rsrc_t rc, unsigned vo, float scale, std::integer_sequence<int, Is...>) { unsigned so = 0; (epi_store<Is>(rc, vo, so, scale), ...); }
template <int I> DI void epi_store_q(rsrc_t rc, unsigned vo, float scale) {
  constexpr int r = I & 3, fx = I >> 2;
  const f32x4 v = {accrd<AI(0, fx, 0, 0) + r>() * scale, accrd<AI(0, fx, 0, 1) + r>() * scale, accrd<AI(0, fx, 0, 2) + r>() * scale, accrd<AI(0, fx, 0, 3) + r>() * scale};
  __builtin_amdgcn_raw_buffer_store_b128(__builtin_bit_cast(u32x4, v), rc, vo, ((fx * 16 + r) * N) * 4, AUX_NT);
  if constexpr (r & 1) GAP();
}
template <int... Is> DI void epilogue_q_(rsrc_t rc, unsigned vo, float scale, std::integer_sequence<int, Is...>) { (epi_store_q<Is>(rc, vo, scale), ...); }

template <int P, bool ZERO, int RS, int DS, bool PICK, int Mi> DI void gap_q(Ctx& c, unsigned sb, unsigned& so1) {
  constexpr int fx = Mi >> 2, iw = (fx & 1) ? 3 - (Mi & 3) : (Mi & 3);
  mfma<AI(0, fx, 0, iw), ZERO>(c.X[P][fx], c.W[P][iw]);
  if constexpr (Mi < 8) ldsread<(RS & 1) * BLK, Mi>(c, c.X[P ^ 1], RS < 2 ? c.rdx : c.rdx2);
  if constexpr (Mi < 8 && (Mi & 1)) ldsread<(RS & 1) * BLK, Mi / 2>(c, c.W[P ^ 1], RS < 2 ? c.rdw : c.rdw2);
  if constexpr (Mi >= 8 && Mi < 12) ldsread<(RS & 1) * BLK, Mi - 4>(c, c.W[P ^ 1], RS < 2 ? c.rdw : c.rdw2);
  if constexpr (Mi == 0) { so1 = c.x2 + 4096; asm volatile("" : "+s"(so1)); }
  if constexpr (Mi < 8 && !(Mi & 1)) dma<DS * BLK, Mi / 2>(c, c.x2);
  if constexpr (Mi >= 8 && Mi < 12) dma<DS * BLK + 4096, Mi - 8>(c, so1);
  if constexpr (PICK) {
    if constexpr (Mi == 12) pick0(c); if constexpr (Mi == 13) pick1a(c); if constexpr (Mi == 14) pick1b(c);
    if constexpr (Mi == 15) { c.x2 = sb + c.ks; asm volatile("" : "+s"(c.x2)); }
  } else if constexpr (Mi == 12) c.x2 = OOB;
  if constexpr (Mi == 15) { __builtin_amdgcn_s_waitcnt(WAITCNT(16, 0)); __builtin_amdgcn_s_barrier(); }
  GAP();
}
template <int P, bool ZERO, int RS, int DS, bool PICK, int... Ms> DI void step_q_(Ctx& c, unsigned sb, std::integer_sequence<int, Ms...>) { unsigned so1 = 0; (gap_q<P, ZERO, RS, DS, PICK, Ms>(c, sb, so1), ...); }
template <int P, bool ZERO, int RS, int DS, bool PICK> DI void step_q(Ctx& c, unsigned sb) { step_q_<P, ZERO, RS, DS, PICK>(c, sb, std::make_integer_sequence<int, 16>{}); }

DI void item_of(const Args& a, int i, bool& isx, int& p, int& j) {
  const unsigned wd = __builtin_amdgcn_readfirstlane(a.order[i >> 1]), e = (i & 1) ? wd >> 16 : wd & 0xffffu;
  j = e & 3; p = e >> 2; isx = p < 16; p = isx ? p : p - 16;
}
DI void item_linear(int i, bool& isx, int& p, int& j) { isx = i < 64; j = i & 3; p = isx ? i >> 2 : (i - 64) >> 2; }
struct Unit { bool isx; unsigned src, dst, flag; };
template <bool ORDERED> DI Unit unit_of(const Args& a, int u, int rank) {
  Unit r; int p, j;
  if (ORDERED) item_of(a, u >> 3, r.isx, p, j); else item_linear(u >> 3, r.isx, p, j);
  const int sub = u & 7, w = sub >> 2, g = (sub >> 1) & 1, h = sub & 1, ks = j * 8 + rank;
  r.src = r.isx ? (unsigned)((p * 256 + w * 128 + g * 64) * KL + j * 128 + h * 64) : (unsigned)((j * 128 + h * 64) * N + p * 256 + w * 128 + g * 64);
  r.dst = (r.isx ? 0u : XBYTES) + (unsigned)((p * 32 + ks) * 2 + g) * GRP + w * 8192 + h * 1024;
  r.flag = (unsigned)((((r.isx ? p : 16 + p) * 32 + ks) * 8 + sub) * 4);
  r.src = __builtin_amdgcn_readfirstlane(r.src); r.dst = __builtin_amdgcn_readfirstlane(r.dst);
  return r;
}
DI void load_raw(unsigned (&raw)[16], const Unit& u, rsrc_t rx, rsrc_t rw, unsigned vx, unsigned vw) {
  if (u.isx) {
#pragma unroll
    for (int f = 0; f < 4; ++f) { const u32x4 v = __builtin_amdgcn_raw_buffer_load_b128(rx, vx, u.src + f * 16 * KL, 0); raw[4 * f] = v.x; raw[4 * f + 1] = v.y; raw[4 * f + 2] = v.z; raw[4 * f + 3] = v.w; }
  } else {
#pragma unroll
    for (int b = 0; b < 16; ++b) raw[b] = __builtin_amdgcn_raw_buffer_load_b32(rw, vw, u.src + b * N, 0);
  }
}
DI void cook(u32x4 (&out)[4], const unsigned (&raw)[16], bool isx) {
  if (isx) {
#pragma unroll
    for (int f = 0; f < 4; ++f) out[f] = (u32x4){raw[4 * f], raw[4 * f + 1], raw[4 * f + 2], raw[4 * f + 3]};
  } else {
#pragma unroll
    for (int b = 0; b < 4; ++b) {
      const unsigned r0 = raw[4 * b], r1 = raw[4 * b + 1], r2 = raw[4 * b + 2], r3 = raw[4 * b + 3];
      const unsigned t0 = __builtin_amdgcn_perm(r1, r0, 0x05010400u), t1 = __builtin_amdgcn_perm(r1, r0, 0x07030602u);
      const unsigned t2 = __builtin_amdgcn_perm(r3, r2, 0x05010400u), t3 = __builtin_amdgcn_perm(r3, r2, 0x07030602u);
      out[0][b] = __builtin_amdgcn_perm(t2, t0, 0x05040100u); out[1][b] = __builtin_amdgcn_perm(t2, t0, 0x07060302u);
      out[2][b] = __builtin_amdgcn_perm(t3, t1, 0x05040100u); out[3][b] = __builtin_amdgcn_perm(t3, t1, 0x07060302u);
    }
  }
}
constexpr int NU = 2560;
DI unsigned read_call(char* myws) { return __builtin_amdgcn_readfirstlane(__hip_atomic_load((unsigned*)(myws + WS_DONE), __ATOMIC_RELAXED, __HIP_MEMORY_SCOPE_AGENT)) / GRID + 1; }
DI unsigned pusher(const Args& a, char* myws, int wave, int lane) {
  constexpr int NW = NPUSH * 4;
  const int gw = blockIdx.x * 4 + wave;
  const rsrc_t rx = mkrsrc(a.x, M * KL), rw = mkrsrc(a.w, KL * N);
  const int cc = lane & 15, q = lane >> 4;
  const unsigned vx = cc * KL + q * 16, vw = q * 16 * N + cc * 4, vd = lane * 16;
  unsigned raw[16]; u32x4 out[4];
  Unit cur = unit_of<true>(a, gw, a.rank);
  load_raw(raw, cur, rx, rw, vx, vw);
  const unsigned call = read_call(myws);
  unsigned* uq = (unsigned*)(myws + WS_UNITQ);
  if (lane == 0) __hip_atomic_fetch_max(uq, call * 4096u + 2 * NW, __ATOMIC_RELAXED, __HIP_MEMORY_SCOPE_AGENT);
  const rsrc_t ru = mkrsrc(uq, 4); const unsigned vu = lane == 0 ? 0u : OOB;
  rsrc_t rd[NR - 1];
  char* fl = nullptr;
#pragma unroll
  for (int i = 0; i < NR - 1; ++i) {
    const int d = (a.rank + 1 + (i + gw) % 7) & 7;
    char* p = a.ws[0];
#pragma unroll
    for (int q = 1; q < NR; ++q) p = d == q ? a.ws[q] : p;
    rd[i] = mkrsrc(p + WS_STAGE + (size_t)(call & 1) * STAGE_BYTES, STAGE_BYTES);
    fl = lane == i ? p + WS_FLAGS : fl;
  }
  unsigned prev_flag = 0; bool have_prev = false;
  int un = gw + NW;
  for (bool go = true; go;) {
    cook(out, raw, cur.isx);
    go = un < NU;
    const Unit nxt = unit_of<true>(a, go ? un : gw, a.rank);
    load_raw(raw, nxt, rx, rw, vx, vw);
    const unsigned t = __builtin_amdgcn_raw_ptr_buffer_atomic_add_i32(1, ru, vu, 0, 0);
    GAP();
#pragma unroll
    for (int s = 0; s < PWAIT; ++s) __builtin_amdgcn_raw_buffer_store_b128(out[s / 7], rd[s % 7], vd + (s / 7) * 2048, cur.dst, AUX_SYS);
    __builtin_amdgcn_s_waitcnt(WAITCNT(PWAIT, 15));
    asm volatile("" ::: "memory");
    if (have_prev && lane < NR - 1) __hip_atomic_store((unsigned*)(fl + prev_flag), call, __ATOMIC_RELAXED, __HIP_MEMORY_SCOPE_SYSTEM);
    un = __builtin_amdgcn_readfirstlane(t) - call * 4096u;
    GAP();
#pragma unroll
    for (int s = PWAIT; s < 28; ++s) __builtin_amdgcn_raw_buffer_store_b128(out[s / 7], rd[s % 7], vd + (s / 7) * 2048, cur.dst, AUX_SYS);
    GAP();
    prev_flag = cur.flag; have_prev = true; cur = nxt;
  }
  asm volatile("s_waitcnt vmcnt(0)" ::: "memory");
  if (lane < NR - 1) __hip_atomic_store((unsigned*)(fl + prev_flag), call, __ATOMIC_RELAXED, __HIP_MEMORY_SCOPE_SYSTEM);
  return call;
}
DI unsigned local_tiling(const Args& a, char* myws, int wave, int lane) {
  constexpr int NW = (GRID - NPUSH) * 4;
  const int gw = (blockIdx.x - NPUSH) * 4 + wave;
  const rsrc_t rx = mkrsrc(a.x, M * KL), rw = mkrsrc(a.w, KL * N);
  const int cc = lane & 15, q = lane >> 4;
  const unsigned vx = cc * KL + q * 16, vw = q * 16 * N + cc * 4, vd = lane * 16;
  unsigned raw[3][16]; Unit un[3];
#pragma unroll
  for (int i = 0; i < 3; ++i) { const int u = gw + i * NW; un[i] = unit_of<false>(a, u < NU ? u : gw, a.rank); load_raw(raw[i], un[i], rx, rw, vx, vw); }
  const unsigned call = read_call(myws);
  const rsrc_t rd = mkrsrc(myws + WS_STAGE + (size_t)(call & 1) * STAGE_BYTES, STAGE_BYTES);
#pragma unroll
  for (int i = 0; i < 3; ++i) {
    u32x4 out[4]; cook(out, raw[i], un[i].isx);
#pragma unroll
    for (int s = 0; s < 4; ++s) __builtin_amdgcn_raw_buffer_store_b128(out[s], rd, vd + s * 2048, un[i].dst, AUX_SYS);
  }
  asm volatile("s_waitcnt vmcnt(0)" ::: "memory");
  const unsigned f = lane == 0 ? un[0].flag : lane == 1 ? un[1].flag : un[2].flag;
  if (lane < 3) __hip_atomic_store((unsigned*)(myws + WS_FLAGS + f), call, __ATOMIC_RELAXED, __HIP_MEMORY_SCOPE_SYSTEM);
  return call;
}

#define STAMP(i)

DI int draw_more(unsigned* q, unsigned call, int x) {
  const int lane = threadIdx.x & 63;
  for (;;) {
    const unsigned v = lane < 8 ? __hip_atomic_load(q + ((x + lane) & 7) * 32, __ATOMIC_RELAXED, __HIP_MEMORY_SCOPE_AGENT) - call * 1024u : ~0u;
    const unsigned long long open = __ballot(v >= (unsigned)NFULL && v < (unsigned)NITEM);
    if (!open) return -1;
    const int y = (x + __builtin_ctzll(open)) & 7;
    unsigned t = 0;
    if (lane == 0) t = __hip_atomic_fetch_add(q + y * 32, 1u, __ATOMIC_RELAXED, __HIP_MEMORY_SCOPE_AGENT);
    t = __builtin_amdgcn_readfirstlane(t) - call * 1024u;
    if (t < (unsigned)NITEM) return y * 256 + t;
  }
}

__global__ void __launch_bounds__(256, 1) __attribute__((amdgpu_num_vgpr(200))) fused_gemm_gather(Args a) {
  extern __shared__ __attribute__((aligned(16))) char smem[];
  const int tid = threadIdx.x, lane = tid & 63, wave = __builtin_amdgcn_readfirstlane(tid >> 6);
  char* myws = a.ws[0];
#pragma unroll
  for (int q = 1; q < NR; ++q) myws = a.rank == q ? a.ws[q] : myws;
  STAMP(0);
  Ctx c;
  c.lds = (LAS char*)smem;
  for (int i = tid; i < (int)(LDS_BYTES - LDS_RING) / 16; i += 256) *(volatile LAS u32x4*)(c.lds + LDS_RING + i * 16) = (u32x4){0u, 0u, 0u, 0u};
  const unsigned call = __builtin_amdgcn_readfirstlane(blockIdx.x < NPUSH ? pusher(a, myws, wave, lane) : local_tiling(a, myws, wave, lane));
  const float scale = a.sx[0] * a.sw[0];
  const int wr = wave >> 1, wc = wave & 1, fr = lane & 15, fq = lane >> 4, xcd = blockIdx.x & 7;
  unsigned* queue = (unsigned*)(myws + WS_QUEUE);
  c.rq = mkrsrc(queue, 1024);
  c.rs = mkrsrc(myws + WS_STAGE + (size_t)(call & 1) * STAGE_BYTES, STAGE_BYTES);
  c.rf = mkrsrc(myws + WS_FLAGS, FLAG_BYTES);
  c.tbase = __builtin_amdgcn_s_memrealtime() - a.kphase; c.kclk = a.useclk ? a.kclk : 0;
  c.call = call; c.tmo = (unsigned*)(myws + WS_TMO); c.wave = wave; c.snap = 0; c.ti = 0; c.npos = 0; c.ptr = 0; c.qmode = 0; c.x2 = 0; c.w2 = 0; c.pos = 0; c.kn = 0; c.av = 0; c.quota = 0; c.px = 0; c.pw = 0; c.fv = 0; c.ks = 0;
  const unsigned vo = (fq * 4 * N + fr * 4) * 4;
  asm volatile("" ::: ACLOB);
  int epos = -1;
  const bool gemm = blockIdx.x >= NPUSH;
  bool full_todo = gemm;
  if (tid == 192 && gemm) __hip_atomic_fetch_max(queue + xcd * 32, call * 1024u + 8 + GPX, __ATOMIC_RELAXED, __HIP_MEMORY_SCOPE_AGENT);
  __builtin_amdgcn_s_waitcnt(WAITCNT(0, 0));
  __builtin_amdgcn_s_barrier();
  if (!gemm) STAMP(1);
  if (wave == 3) *(volatile LAS int*)(c.lds + LDS_TICKET) = gemm ? xcd * 256 + 255 : draw_more(queue, call, xcd);
#pragma unroll 1
  for (;;) {
    __builtin_amdgcn_s_waitcnt(WAITCNT(63, 0));
    __builtin_amdgcn_s_barrier();
    const int qid = __builtin_amdgcn_readfirstlane(*(volatile LAS int*)(c.lds + LDS_TICKET));
    const int t = qid & 255, q = t == 255 ? (int)(blockIdx.x - NPUSH) >> 3 : t < NFULL + (128 - NFULL) * 4 ? t + 3 * NFULL : t - (NFULL + (128 - NFULL) * 4) + GPX;
    const int hx = (q >> 1) & 1, hw = q & 1;
    int mt, nt; tile_of(qid >> 8, q >> 2, mt, nt);
    const unsigned sb = __builtin_amdgcn_readfirstlane((wave < 2 ? mt * PANEL + hx * 8192 : XBYTES + nt * PANEL + hw * 8192) + (wave & 1) * GRP);
    c.rdx = wr * 8192 + lane * 16; c.rdw = 16384 + wc * 8192 + lane * 16; c.rdx2 = c.rdx + 65536; c.rdw2 = c.rdw + 65536;
    asm volatile("" : "+v"(c.rdx2), "+v"(c.rdw2));
    c.voff = lane * 16; c.m0w = wave * 8192;
#define DMA8(S) do { pick0(c); pick1(c); dma4<(S) * BLK>(c, sb + c.ks); dma4<(S) * BLK + 4096>(c, sb + c.ks + 4096); __builtin_amdgcn_s_barrier(); } while (0)
    if (qid >= 0) {
      open_tile(c, q >> 2, qid >> 8); c.qmode = 1; c.snap = 0; c.ti = full_todo ? 0 : 8;
      if (epos < 0) { DMA8(0); DMA8(1); DMA8(2); DMA8(3); }
    }
    if (epos >= 0) {
      int mt3, nt3; tile_of(xcd, epos, mt3, nt3);
      const rsrc_t rc = mkrsrc((char*)a.out + ((size_t)(mt3 * 256 + wr * 128) * N + nt3 * 256 + wc * 128) * 4, 128u * N * 4);
      const float sc = *(volatile LAS unsigned*)(c.lds + LDS_DEAD) ? __builtin_nanf("") : scale;
      asm volatile("s_nop 15" ::: ACLOB);
      epilogue_(rc, vo, sc, std::make_integer_sequence<int, 64>{});
      GAP();
      STAMP(10);
      epos = -1;
      if (qid >= 0) { DMA8(0); DMA8(1); DMA8(2); DMA8(3); }
    }
    if (qid < 0) break;
    pick0(c); pick1(c); c.x2 = sb + c.ks;
    __builtin_amdgcn_s_waitcnt(WAITCNT(24, 15));
    __builtin_amdgcn_s_barrier();
    ldsread8<0>(c, c.X[0], c.rdx); ldsread8<0>(c, c.W[0], c.rdw);
    __builtin_amdgcn_s_waitcnt(WAITCNT(16, 0));
    __builtin_amdgcn_s_barrier();
    GAP();
    step_q<0, true, 1, 0, true>(c, sb); step_q<1, false, 2, 1, true>(c, sb); step_q<0, false, 3, 2, true>(c, sb); step_q<1, false, 0, 3, true>(c, sb);
#pragma unroll 1
    for (int k = 4; k < 24; k += 4) { step_q<0, false, 1, 0, true>(c, sb); step_q<1, false, 2, 1, true>(c, sb); step_q<0, false, 3, 2, true>(c, sb); step_q<1, false, 0, 3, true>(c, sb); }
    step_q<0, false, 1, 0, true>(c, sb); step_q<1, false, 2, 1, true>(c, sb); step_q<0, false, 3, 2, true>(c, sb); step_q<1, false, 0, 3, false>(c, sb);
    step_q<0, false, 1, 0, false>(c, sb); step_q<1, false, 2, 1, false>(c, sb); step_q<0, false, 3, 2, false>(c, sb); step_q<1, false, 0, 3, false>(c, sb);
    __builtin_amdgcn_s_waitcnt(WAITCNT(0, 0));
    __builtin_amdgcn_s_barrier();
    if (full_todo) {
      c.rdx = wr * 8192 + lane * 16; c.rdw = wc * 8192 + lane * 16; c.rdx2 = c.rdx + 65536; c.rdw2 = c.rdw + 65536;
      asm volatile("" : "+v"(c.rdx2), "+v"(c.rdw2));
      c.voff = wave * 4096 + lane * 16; c.m0w = wave * 4096;
      epos = 8 + ((blockIdx.x - NPUSH) >> 3);
      open_tile(c, epos); c.qmode = 0; c.snap = 0; c.ti = 1;
      unsigned x0, w0, x1, w1;
      pick0(c); pick1(c); pick2(c, x0, w0); __builtin_amdgcn_s_barrier();
      pick0(c); pick1(c); pick2(c, x1, w1); __builtin_amdgcn_s_barrier();
      pick0(c); pick1(c); pick2(c, c.x2, c.w2);
      dma4<0 * GRP>(c, x0); dma4<1 * GRP>(c, w0); dma4<2 * GRP>(c, w0 + GRP); dma4<3 * GRP>(c, x0 + GRP);
      dma4<4 * GRP>(c, x1); dma4<5 * GRP>(c, w1 + GRP); dma4<6 * GRP>(c, w1); dma4<7 * GRP>(c, x1 + GRP);
    }
    { const rsrc_t rc = mkrsrc((char*)a.out + ((size_t)(mt * 256 + hx * 128 + wr * 64) * N + nt * 256 + hw * 128 + wc * 64) * 4, 64u * N * 4);
      const float sc = *(volatile LAS unsigned*)(c.lds + LDS_DEAD) ? __builtin_nanf("") : scale;
      asm volatile("s_nop 15" ::: ACLOB);
      epilogue_q_(rc, vo, sc, std::make_integer_sequence<int, 16>{}); }
    if (!full_todo) {
      if (wave == 3) *(volatile LAS int*)(c.lds + LDS_TICKET) = draw_more(queue, call, xcd);
      continue;
    }
    full_todo = false;
    c.x2 = __builtin_amdgcn_readfirstlane(c.x2); c.w2 = __builtin_amdgcn_readfirstlane(c.w2);
    __builtin_amdgcn_s_waitcnt(WAITCNT(24, 15));
    __builtin_amdgcn_s_barrier();
    ldsread8<0 * GRP>(c, c.X[0], c.rdx); ldsread8<1 * GRP>(c, c.W[0], c.rdw);
    __builtin_amdgcn_s_waitcnt(WAITCNT(20, 0));
    __builtin_amdgcn_s_barrier();
    dma4<0 * GRP>(c, c.x2);
    GAP();
#pragma unroll 1
    for (;;) {
      two_steps<true>(c);
      int n = 15;
#pragma unroll 1
      do { two_steps_tail(c); two_steps<false>(c); asm volatile("" : "+s"(n)); } while (--n);
      two_steps_tail(c);
      if (c.pos >= NFULL) break;
      int mt2, nt2; tile_of(xcd, epos, mt2, nt2);
      const rsrc_t rc = mkrsrc((char*)a.out + ((size_t)(mt2 * 256 + wr * 128) * N + nt2 * 256 + wc * 128) * 4, 128u * N * 4);
      const float sc = *(volatile LAS unsigned*)(c.lds + LDS_DEAD) ? __builtin_nanf("") : scale;
      asm volatile("s_nop 15" ::: ACLOB);
      epilogue_(rc, vo, sc, std::make_integer_sequence<int, 64>{});
      GAP();
      epos = c.pos;
    }
    __builtin_amdgcn_s_waitcnt(WAITCNT(0, 0));
    if (wave == 3) *(volatile LAS int*)(c.lds + LDS_TICKET) = c.pos < NITEM ? xcd * 256 + c.pos : draw_more(queue, call, xcd);
  }
  if (tid == 0) __hip_atomic_fetch_add((unsigned*)(myws + WS_DONE), 1u, __ATOMIC_RELAXED, __HIP_MEMORY_SCOPE_AGENT);
}

}

struct Tune { float slope, xlead, xslope; int rot, cyc, kphase, kclk, useclk, grp; };
static Tune g_tune = {0.0625f, 0.0f, 0.0625f, 0, 0, 2200, 546, 1, 16};
static void build_order(Args& a) {
  const Tune& t = g_tune;
  float key[320]; unsigned short e[320];
  for (int i = 0; i < 320; ++i) {
    const int P = i >> 2, s = i & 3;
    if (P < 16) { key[i] = s + ((P + t.rot * a.rank) & 15) * t.xslope - t.xlead; e[i] = (unsigned short)i; }
    else {
      const int p = P - 16 - 4; const float off = p < 0 ? 0.f : (p % t.grp) * t.slope;
      key[i] = p < 0 ? s - t.xlead + 0.2f * (p + 4) : (p / t.grp) * 4 + 4 + s + off + 0.01f;
      e[i] = (unsigned short)((P << 2) | ((s + (t.cyc ? (int)(off + 0.5f) : 0)) & 3));
    }
  }
  for (int i = 1; i < 320; ++i) { const float k = key[i]; const unsigned short v = e[i]; int q = i - 1; while (q >= 0 && key[q] > k) { key[q + 1] = key[q]; e[q + 1] = e[q]; --q; } key[q + 1] = k; e[q + 1] = v; }
  for (int i = 0; i < 160; ++i) a.order[i] = e[2 * i] | ((unsigned)e[2 * i + 1] << 16);
  a.kphase = t.kphase; a.kclk = t.kclk; a.useclk = t.useclk;
}

extern "C" void kernel_launch(int rank, int nranks, ncclComm_t comm, hipStream_t stream,
                              void* const* d_in, const int* in_sizes, int n_in,
                              void* d_out, int out_size, const int* in_dtypes, int out_dtype,
                              void* const* d_out_all, void* const* d_ws_all, size_t ws_bytes) {
  if (nranks != NR || (unsigned)rank >= (unsigned)NR || n_in != 4 || in_sizes[0] != M * KL || in_sizes[1] != KL * N || out_size != M * N || ws_bytes < WS_NEED) {
    fprintf(stderr, "kernel_launch: unexpected problem shape\n");
    return;
  }
  Args a;
  a.x = (const unsigned char*)d_in[0]; a.w = (const unsigned char*)d_in[1]; a.sx = (const float*)d_in[2]; a.sw = (const float*)d_in[3];
  a.out = (float*)d_out; a.rank = rank; a.dbg = nullptr;
  for (int p = 0; p < NR; ++p) a.ws[p] = (char*)d_ws_all[p];
  build_order(a);
  HIP_CHECK(hipFuncSetAttribute((const void*)fused_gemm_gather, hipFuncAttributeMaxDynamicSharedMemorySize, LDS_BYTES));
  hipLaunchKernelGGL(fused_gemm_gather, dim3(GRID), dim3(256), LDS_BYTES, stream, a);
  HIP_CHECK(hipGetLastError());
}
```

```cpp
#include <hip/hip_runtime.h>
#include <rccl/rccl.h>
#include <cstdio>
#include <utility>

#define HIP_CHECK(expr) do { hipError_t _e = (expr); if (_e != hipSuccess) { \
    fprintf(stderr, "HIP error %s at %s:%d\n", hipGetErrorString(_e), __FILE__, __LINE__); \
    return; } } while (0)

namespace {

typedef unsigned u32x2 __attribute__((ext_vector_type(2)));
typedef unsigned u32x4 __attribute__((ext_vector_type(4)));
typedef __amdgpu_buffer_rsrc_t rsrc_t;
#define LAS __attribute__((address_space(3)))
#define DI __device__ __forceinline__
#define GAP() __builtin_amdgcn_sched_barrier(0)
#define RFL(x) __builtin_amdgcn_readfirstlane(x)

constexpr int M = 4096, N = 16384, K = 4096, NR = 8, KL = K / NR;
constexpr unsigned GRP = 16384, STRIP = 32 * GRP, STAGE_BYTES = 160 * STRIP;
constexpr int GRID = 256, NPUSH = 16, NU = 2560;
constexpr size_t WS_DONE = 0, WS_TMO = 128, WS_UNITQ = 256, WS_FLAGS = 4096, WS_STAGE = 1u << 20;
constexpr unsigned FLAG_BYTES = 160 * 32 * 16;
constexpr size_t WS_NEED = WS_STAGE + 2 * (size_t)STAGE_BYTES;
constexpr unsigned LDS_RING = 8 * GRP, LDS_DEAD = LDS_RING, LDS_SEL = LDS_RING + 16, LDS_SNAP = LDS_RING + 64, LDS_BYTES = LDS_SNAP + 4 * 1024;
constexpr unsigned long long DEADLINE = 40000000ull;
constexpr int AUX_SYS = 17, AUX_OUT = 2, AUX_LAST = 18;
constexpr unsigned PITCH = N * 4;
constexpr unsigned OOB = 0x40000000u;

struct Args {
  const unsigned char* x; const unsigned char* w; const float* sx; const float* sw; float* out;
  char* ws[NR]; int rank;
  unsigned order[320];
};

constexpr int WAITCNT(int vm, int lgkm) { return (vm & 15) | (7 << 4) | ((lgkm & 15) << 8) | ((vm >> 4) << 14); }
DI rsrc_t mkrsrc(const void* p, unsigned bytes) {
  const unsigned long long v = (unsigned long long)p;
  const unsigned long long u = ((unsigned long long)(unsigned)RFL((unsigned)(v >> 32)) << 32) | (unsigned)RFL((unsigned)v);
  return __builtin_amdgcn_make_buffer_rsrc((void*)u, 0, bytes, 0x00020000);
}

struct Ctx {
  LAS char* lds;
  unsigned rdx, rdw, rdx2, rdw2;
  unsigned voff, m0w;
  rsrc_t rs;
  unsigned x2, a2, b2, h2;
  rsrc_t rf; unsigned fv; unsigned call; unsigned* tmo; int wave, team, mt;
  unsigned av, kn, ks, wb, full, seen, snapv, notme, snapdst, ta, tb, tf, kv, selA, selB; int quota, ph, lead; unsigned px, pa, pb;
  u32x4 snap; unsigned long long m64;
  rsrc_t rc; unsigned vo; float sc;
};

constexpr int AI(int gx, int fx, int gw, int iw) { return (((gx * 4 + fx) * 2 + gw) * 4 + iw) * 4; }
constexpr int XR(int g) { return 64 + g * 32; }
constexpr int WR(int g) { return 128 + g * 32; }
constexpr int NSTASH = 15;
constexpr int ER(int k) { return 192 + (k % NSTASH) * 4; }
template <int A, bool ZERO, int X, int W> DI void mfma() {
  if constexpr (ZERO) asm volatile("v_mfma_f32_16x16x128_f8f6f4 a[%c0:%c0+3], v[%c1:%c1+7], v[%c2:%c2+7], 0 cbsz:1 blgp:1" :: "i"(A), "i"(X), "i"(W));
  else asm volatile("v_mfma_f32_16x16x128_f8f6f4 a[%c0:%c0+3], v[%c1:%c1+7], v[%c2:%c2+7], a[%c0:%c0+3] cbsz:1 blgp:1" :: "i"(A), "i"(X), "i"(W));
}
template <int A, int V> DI void accmul2(float s) {
  asm volatile("v_accvgpr_read_b32 v[%c1], a[%c0]\n\tv_accvgpr_read_b32 v[%c1+1], a[%c0+4]\n\tv_mul_f32 v[%c1], %2, v[%c1]\n\tv_mul_f32 v[%c1+1], %2, v[%c1+1]" :: "i"(A), "i"(V), "s"(s));
}
template <int V, int AUX> DI void store16(rsrc_t rc, unsigned vo, unsigned so) {
  if constexpr (AUX == AUX_OUT) asm volatile("buffer_store_dwordx4 v[%c3:%c3+3], %0, %1, %2 offen nt" :: "v"(vo), "s"(rc), "s"(so), "i"(V) : "memory");
  else asm volatile("buffer_store_dwordx4 v[%c3:%c3+3], %0, %1, %2 offen sc1 nt" :: "v"(vo), "s"(rc), "s"(so), "i"(V) : "memory");
}
template <int OFF, int I, int R> DI void ldsread(unsigned rd) {
  asm volatile("ds_read_b128 v[%c1:%c1+3], %0 offset:%c2" :: "v"(rd), "i"(R + 4 * I), "i"(OFF + I * 1024));
}
template <int OFF, int R, int... Is> DI void ldsread8_(unsigned rd, std::integer_sequence<int, Is...>) { (ldsread<OFF, Is, R>(rd), ...); }
template <int OFF, int R> DI void ldsread8(unsigned rd) { ldsread8_<OFF, R>(rd, std::make_integer_sequence<int, 8>{}); }
template <int LOFF, int I> DI void dma(const Ctx& c, unsigned soff) {
  __builtin_amdgcn_raw_ptr_buffer_load_lds(c.rs, (LAS void*)(c.lds + LOFF + c.m0w), 16, c.voff, soff, I * 1024, 0);
}
template <int LOFF> DI void dma4(const Ctx& c, unsigned soff) { dma<LOFF, 0>(c, soff); dma<LOFF, 1>(c, soff); dma<LOFF, 2>(c, soff); dma<LOFF, 3>(c, soff); }

__host__ __device__ inline int strip_of(int h, int team) { return h < 0 || h > 8 ? -1 : h == 0 ? ((team & 2) ? -1 : (team >> 2) * 2 + (team & 1)) : 8 + 15 * (h - 1) + team; }
__host__ __device__ inline int lead_of(int team) { return (team & 2) ? ((team & 1) ? 6 : 3) : 0; }
DI int col_of(int s, int w) { return s * 128 + w * 64; }

DI void make_tables(Ctx& c) {
  const int lane = threadIdx.x & 63, h = lane & 15, part = lane >> 4;
  const int sa = strip_of(h & ~1, c.team), sb = strip_of((h - 1) | 1, c.team);
  c.ta = sa < 0 ? OOB : (32 + sa) * STRIP; c.tb = sb < 0 ? OOB : (32 + sb) * STRIP;
  c.tf = (unsigned)((sa < 0 ? 2 * c.mt : 32 + sa) * 512) | (unsigned)((sb < 0 ? 2 * c.mt : 32 + sb) * 512) << 15;
  c.kv = ((h - c.lead) & 31) * 16 + (part < 2 ? (2 * c.mt + part) * 512 : 0);
  c.selA = part == 2 ? 0x1ffffu : 0u; c.selB = part == 3 ? 0x1ffffu : 0u;
}
DI unsigned flag_voff(const Ctx& c, int h) {
  const unsigned f = __builtin_amdgcn_readlane(c.tf, h);
  return (c.kv ^ ((h & 1) << 8)) + ((f & 0x1fe00u) & c.selA) + (((f >> 15) & 0x1fe00u) & c.selB);
}
DI unsigned long long cmp_flags(const Ctx& c, const u32x4& a) {
  const unsigned cl = c.call;
  return __ballot((a.x >= cl) & (a.y >= cl) & (a.z >= cl) & (a.w >= cl));
}
DI unsigned fold64(unsigned long long m) { const unsigned t = (unsigned)m & (unsigned)(m >> 32); return t & (t >> 16); }
DI unsigned fold_flags(const Ctx& c, const u32x4& a) { return fold64(cmp_flags(c, a)); }
DI void open_half(Ctx& c, int h) {
  const unsigned over = h > 9 ? 0xffffu : 0u;
  c.ph = h; c.kn = over; c.av = over; c.fv = c.snapv; c.wb ^= 16; c.full = 0xffffu;
  c.pa = __builtin_amdgcn_readlane(c.ta, h); c.pb = __builtin_amdgcn_readlane(c.tb, h);
  asm volatile("" : "+s"(c.pa), "+s"(c.pb), "+s"(c.wb), "+s"(c.kn), "+s"(c.av), "+s"(c.ph));
}
DI void turn(Ctx& c) {
  if (c.full == 0xffffu) { c.snapv = flag_voff(c, c.ph + 1) | c.notme; c.quota = 2; c.full = ~0u; }
  else { open_half(c, c.ph + 1); c.quota = 14; }
}
DI void starve(Ctx& c) {
  if (c.wave == 3) {
    const unsigned long long t0 = __builtin_amdgcn_s_memrealtime();
    unsigned m;
    for (;;) {
      m = fold_flags(c, __builtin_amdgcn_raw_buffer_load_b128(c.rf, c.fv, 0, AUX_SYS));
      if (m & ~c.kn) break;
      if (RFL(__hip_atomic_load(c.tmo, __ATOMIC_RELAXED, __HIP_MEMORY_SCOPE_AGENT)) == c.call || __builtin_amdgcn_s_memrealtime() - t0 > DEADLINE) {
        __hip_atomic_store(c.tmo, c.call, __ATOMIC_RELAXED, __HIP_MEMORY_SCOPE_AGENT);
        *(volatile LAS unsigned*)(c.lds + LDS_DEAD) = 1u;
        m = 0xffffu; break;
      }
      __builtin_amdgcn_s_sleep(2);
    }
    *(volatile LAS unsigned*)(c.lds + LDS_SEL) = m;
  }
  __builtin_amdgcn_s_waitcnt(WAITCNT(63, 0));
  __builtin_amdgcn_s_barrier();
  const unsigned m = RFL(*(volatile LAS unsigned*)(c.lds + LDS_SEL));
  c.av = m & ~c.kn; c.kn |= m;
}
template <int P> DI void snap_read(Ctx& c) { c.snap = *(volatile LAS u32x4*)(c.lds + LDS_SNAP + P * 1024 + (threadIdx.x & 63) * 16); }
DI void snap_cmp(Ctx& c) { c.m64 = cmp_flags(c, c.snap); asm volatile("" : "+s"(c.m64)); }
DI void snap_fold(Ctx& c) { c.seen = fold64(c.m64); asm volatile("" : "+s"(c.seen)); }
DI void pick0(Ctx& c) { if (__builtin_expect(c.quota == 0, 0)) turn(c); c.quota -= 1; asm volatile("" : "+s"(c.quota)); }
DI void pick0s(Ctx& c) { if (__builtin_expect(c.av == 0, 0)) starve(c); }
DI void pick0m(Ctx& c) { c.av |= c.seen & ~c.kn; c.kn |= c.seen; pick0s(c); }
template <int P> DI void snap_issue(Ctx& c) {
  unsigned so = c.kn == c.full ? OOB : 0u; asm volatile("" : "+s"(so));
  __builtin_amdgcn_raw_ptr_buffer_load_lds(c.rf, (LAS void*)(c.lds + c.snapdst + P * 1024), 16, c.snapv, so, 0, AUX_SYS);
}
DI void pick1(Ctx& c) { c.ks = (__builtin_ctz(c.av) + c.wb) & 31; asm volatile("" : "+s"(c.ks)); }
DI void pick2(Ctx& c) { c.av &= c.av - 1; c.ks *= GRP; asm volatile("" : "+s"(c.ks), "+s"(c.av)); }
DI void pick3(Ctx& c) { c.x2 = c.px + c.ks; c.a2 = c.pa + c.ks; c.b2 = c.pb + c.ks; c.h2 = c.x2 + STRIP; asm volatile("" : "+s"(c.x2), "+s"(c.a2), "+s"(c.b2), "+s"(c.h2)); }

constexpr short EOP[2][7][17] = {{{0, 0, 0, 0, 0, 0, 0, 0, 0, 0, 0, 0, 0, 0, 0, 0, 0}, {0, 3, 4, 8, 9, 12, 16, 19, 21, 23, 27, 30, 34, 35, 35, 37, 38}, {38, 39, 44, 44, 48, 51, 51, 56, 58, 61, 66, 71, 75, 81, 85, 90, 94}, {94, 97, 101, 104, 107, 109, 113, 116, 118, 120, 123, 127, 130, 134, 137, 141, 142}, {142, 145, 149, 152, 155, 157, 161, 164, 166, 168, 171, 175, 178, 182, 185, 189, 190}, {190, 193, 194, 197, 198, 199, 202, 205, 207, 209, 212, 215, 218, 220, 220, 221, 222}, {222, 223, 227, 227, 232, 234, 234, 238, 240, 243, 248, 252, 256, 256, 256, 256, 256}}, {{0, 3, 6, 9, 12, 15, 18, 21, 23, 25, 28, 31, 34, 38, 41, 45, 46}, {46, 49, 50, 53, 54, 55, 58, 61, 63, 65, 68, 71, 74, 76, 76, 77, 78}, {78, 79, 83, 83, 87, 89, 89, 93, 95, 98, 102, 106, 110, 115, 118, 123, 126}, {126, 129, 133, 136, 139, 141, 145, 148, 150, 152, 155, 159, 162, 166, 169, 173, 174}, {174, 178, 182, 186, 190, 192, 196, 200, 202, 204, 208, 212, 216, 221, 224, 229, 230}, {230, 233, 234, 236, 237, 238, 241, 244, 245, 248, 250, 253, 256, 256, 256, 256, 256}, {256, 256, 256, 256, 256, 256, 256, 256, 256, 256, 256, 256, 256, 256, 256, 256, 256}}};
constexpr signed char EST[2][11][16] = {{{-1, -1, -1, -1, -1, -1, -1, -1, -1, -1, -1, -1, -1, -1, -1, -1}, {-1, -1, -1, -1, -1, -1, -1, 0, -1, -1, -1, -1, -1, -1, -1, 1}, {-1, -1, -1, -1, 2, -1, -1, 3, -1, -1, -1, -1, -1, 4, -1, -1}, {-1, -1, -1, -1, 5, -1, -1, 6, -1, -1, -1, -1, -1, 7, -1, 8}, {-1, -1, -1, -1, 9, -1, -1, 10, -1, -1, -1, -1, -1, 11, -1, 12}, {-1, -1, -1, -1, 13, -1, -1, 14, -1, -1, -1, -1, -1, -1, -1, 15}, {-1, -1, -1, -1, 16, -1, -1, 17, -1, -1, -1, -1, -1, 18, -1, -1}, {-1, -1, -1, -1, 19, -1, -1, 20, -1, -1, -1, -1, -1, 21, -1, 22}, {-1, -1, -1, -1, 23, -1, -1, 24, -1, -1, -1, -1, -1, 25, -1, 26}, {-1, -1, -1, -1, 27, -1, -1, 28, -1, -1, -1, -1, -1, -1, -1, 29}, {-1, -1, -1, -1, 30, -1, -1, 31, -1, -1, -1, -1, -1, -1, -1, -1}}, {{-1, -1, -1, -1, -1, -1, -1, 0, -1, -1, -1, -1, -1, -1, -1, 1}, {-1, -1, -1, -1, 2, -1, -1, 3, -1, -1, -1, -1, -1, -1, -1, 4}, {-1, -1, -1, -1, 5, -1, -1, 6, -1, -1, -1, -1, -1, 7, -1, -1}, {-1, -1, -1, -1, 8, -1, -1, 9, -1, -1, -1, -1, -1, 10, -1, 11}, {-1, -1, -1, -1, 12, -1, -1, 13, -1, -1, -1, -1, -1, 14, -1, 15}, {-1, -1, -1, -1, 16, -1, -1, 17, -1, -1, -1, -1, -1, -1, -1, 18}, {-1, -1, -1, -1, 19, -1, -1, 20, -1, -1, -1, -1, -1, 21, -1, -1}, {-1, -1, -1, -1, 22, -1, -1, 23, -1, -1, -1, -1, -1, 24, -1, -1}, {-1, -1, -1, -1, 25, -1, -1, 26, -1, -1, -1, -1, -1, 27, -1, -1}, {-1, -1, -1, -1, 28, -1, -1, 29, -1, -1, -1, -1, -1, -1, -1, -1}, {-1, -1, -1, -1, 30, -1, -1, 31, -1, -1, -1, -1, -1, -1, -1, -1}}};
template <int S, int O> DI void e_op(const Ctx& c) {
  constexpr bool rd = O < 2 || (O < 254 && !(O & 1));
  constexpr int e = O < 2 ? O : O >= 254 ? O - 128 : rd ? (O + 2) / 2 : (O - 3) / 2, k = e >> 2, i = e & 3;
  if constexpr (rd) asm volatile("v_accvgpr_read_b32 v[%c0], a[%c1]" :: "i"(ER(k) + i), "i"(AI(k >> 4, (k >> 2) & 3, S, i) + (k & 3)));
  else asm volatile("v_mul_f32 v[%c0], %1, v[%c0]" :: "i"(ER(k) + i), "s"(c.sc));
}
template <int S, int H> DI void e_read(const Ctx& c) { constexpr int k = H >> 1, i = 2 * (H & 1); accmul2<AI(k >> 4, (k >> 2) & 3, S, i) + (k & 3), ER(k) + i>(c.sc); }
template <int KU, int AUX> DI void e_store(const Ctx& c) {
  unsigned so = ((KU >> 4) * 128 + ((KU >> 2) & 3) * 16 + (KU & 3)) * PITCH; asm volatile("" : "+s"(so));
  store16<ER(KU), AUX>(c.rc, c.vo, so);
}
template <int S, int LO, int... Is> DI void e_ops(const Ctx& c, std::integer_sequence<int, Is...>) { (e_op<S, LO + Is>(c), ...); }
template <int S, int T, int G> DI void e_gap(const Ctx& c) {
  if constexpr (T >= 0) {
    if constexpr (EST[S][T][G] >= 0) e_store<EST[S][T][G], AUX_OUT>(c);
    if constexpr (T < 7) e_ops<S, EOP[S][T][G]>(c, std::make_integer_sequence<int, EOP[S][T][G + 1] - EOP[S][T][G]>{});
  }
}
template <int KU> DI void e_fin(const Ctx& c) { e_read<0, 2 * KU>(c); e_read<0, 2 * KU + 1>(c); e_store<KU, AUX_LAST>(c); GAP(); }
template <int... Ks> DI void e_finish_(const Ctx& c, std::integer_sequence<int, Ks...>) { (e_fin<Ks>(c), ...); }

template <int GX, int GW, bool ZERO, bool RISX, int RG, int RSLOT, int DSLOT, int DSEL, int ES, int ET, int Mi> DI void gap(Ctx& c, unsigned& so) {
  constexpr int fx = Mi >> 2, iw = (fx & 1) ? 3 - (Mi & 3) : (Mi & 3);
  mfma<AI(GX, fx, GW, iw), ZERO, XR(GX) + 8 * fx, WR(GW) + 8 * iw>();
  constexpr int R0 = 0, D0 = 8;
  if constexpr (Mi >= R0 && Mi < R0 + 8) { if constexpr (RISX) ldsread<(RSLOT & 3) * GRP, Mi - R0, XR(RG)>(RSLOT < 4 ? c.rdx : c.rdx2); else ldsread<(RSLOT & 3) * GRP, Mi - R0, WR(RG)>(RSLOT < 4 ? c.rdw : c.rdw2); }
  if constexpr (Mi >= D0 && Mi < D0 + 4) dma<DSLOT * GRP, Mi - D0>(c, DSEL == 0 ? c.a2 : DSEL == 1 ? c.b2 : DSEL == 2 ? c.h2 : c.x2);
  if constexpr (DSLOT % 4 == 2 && Mi == 13) snap_read<DSLOT / 4>(c);
  if constexpr (DSEL == 2) {
    if constexpr (Mi == 1) snap_cmp(c); if constexpr (Mi == 3) snap_fold(c);
    if constexpr (Mi == 12) pick0(c); if constexpr (Mi == 13) pick0m(c); if constexpr (Mi == 14) pick1(c);
  }
  if constexpr (DSEL == 3) { if constexpr (Mi == 0) pick2(c); if constexpr (Mi == 2) pick3(c); if constexpr (Mi == 5) snap_issue<DSLOT == 0>(c); }
  e_gap<ES, ET, Mi>(c);
  if constexpr (Mi == 15) {
    __builtin_amdgcn_s_waitcnt(WAITCNT(24, 0));
    __builtin_amdgcn_s_barrier();
  }
  GAP();
}
template <int GX, int GW, bool ZERO, bool RISX, int RG, int RSLOT, int DSLOT, int DSEL, int ES, int ET, int... Ms> DI void quad_(Ctx& c, std::integer_sequence<int, Ms...>) {
  unsigned so = 0;
  (gap<GX, GW, ZERO, RISX, RG, RSLOT, DSLOT, DSEL, ES, ET, Ms>(c, so), ...);
}
template <int GX, int GW, bool ZERO, bool RISX, int RG, int RSLOT, int DSLOT, int DSEL, int ES, int ET, int NG = 16> DI void quad(Ctx& c) {
  quad_<GX, GW, ZERO, RISX, RG, RSLOT, DSLOT, DSEL, ES, ET>(c, std::make_integer_sequence<int, NG>{});
}
template <int EPI, int S> DI void two_steps(Ctx& c) {
  constexpr bool ZA = EPI == 2 && S == 0, ZB = EPI == 2 && S == 1;
  quad<0, 0, ZA, false, 1, 2, 1, 0, S, EPI == 2 ? 3 : -1>(c);
  quad<0, 1, ZB, true, 1, 3, 2, 1, S, EPI == 2 ? 4 : -1>(c);
  quad<1, 1, ZB, true, 0, 4, 3, 2, S, EPI == 2 ? 5 : -1>(c);
  quad<1, 0, ZA, false, 1, 5, 4, 3, S, EPI == 2 ? 6 : -1>(c);
  quad<0, 1, false, false, 0, 6, 5, 1, S, EPI == 2 ? 7 : -1>(c);
  quad<0, 0, false, true, 1, 7, 6, 0, S, EPI == 2 ? 8 : EPI == 1 ? 0 : -1>(c);
  quad<1, 0, false, true, 0, 0, 7, 2, S, EPI == 2 ? 9 : EPI == 1 ? 1 : -1>(c);
  quad<1, 1, false, false, 0, 1, 0, 3, S, EPI == 2 ? 10 : EPI == 1 ? 2 : -1, EPI == 1 ? 16 : 14>(c);
}
DI void tail(Ctx& c) { unsigned so = 0; gap<1, 1, false, false, 0, 1, 0, 3, 0, -1, 14>(c, so); gap<1, 1, false, false, 0, 1, 0, 3, 0, -1, 15>(c, so); }

struct SoloQ { signed char r0, r1, dw, dh, dl, wait; };
constexpr SoloQ SOLO[32] = {{3, -1, 1, 2, 5, 20}, {4, 6, 3, -1, -1, 20}, {7, -1, -1, 4, 6, 20}, {0, 1, 7, -1, -1, 20}, {2, -1, -1, 0, 1, 20}, {5, 3, 2, -1, -1, 20}, {4, -1, -1, 5, 3, 20}, {6, 7, 4, -1, -1, 20}, {0, -1, -1, 6, 7, 20}, {1, 2, 0, -1, -1, 20}, {5, -1, -1, 1, 2, 20}, {3, 4, 5, -1, -1, 20}, {6, -1, -1, 3, 4, 20}, {7, 0, 6, -1, -1, 20}, {1, -1, -1, 7, 0, 20}, {2, 5, 1, -1, -1, 20}, {3, -1, -1, 2, 5, 20}, {4, 6, 3, -1, -1, 20}, {7, -1, -1, 4, 6, 20}, {0, 1, 7, -1, -1, 20}, {2, -1, -1, 0, 1, 20}, {5, 3, 2, -1, -1, 20}, {4, -1, -1, 5, 3, 20}, {6, 7, 4, -1, -1, 20}, {0, -1, -1, 6, 7, 20}, {1, 2, 0, -1, -1, 20}, {5, -1, -1, 1, -1, 16}, {3, 4, -1, -1, -1, 12}, {6, -1, -1, -1, -1, 4}, {7, 0, -1, -1, -1, 0}, {1, -1, -1, -1, -1, 63}, {-1, -1, -1, -1, -1, 63}};
constexpr short SOP[10][17] = {{126, 128, 129, 131, 132, 134, 135, 137, 138, 140, 141, 143, 144, 146, 147, 149, 150}, {150, 152, 153, 155, 156, 158, 159, 161, 162, 164, 165, 167, 168, 170, 171, 173, 174}, {174, 176, 177, 179, 180, 182, 183, 185, 186, 188, 189, 191, 192, 194, 195, 197, 198}, {198, 200, 201, 203, 204, 206, 207, 209, 210, 212, 213, 215, 216, 218, 219, 221, 222}, {222, 224, 225, 227, 228, 230, 231, 233, 234, 236, 237, 239, 240, 242, 243, 245, 246}, {246, 248, 249, 251, 252, 254, 255, 256, 256, 256, 256, 256, 256, 256, 256, 256, 256}, {256, 256, 256, 256, 256, 256, 256, 256, 256, 256, 256, 256, 256, 256, 256, 256, 256}, {256, 256, 256, 256, 256, 256, 256, 256, 256, 256, 256, 256, 256, 256, 256, 256, 256}, {256, 256, 256, 256, 256, 256, 256, 256, 256, 256, 256, 256, 256, 256, 256, 256, 256}, {256, 256, 256, 256, 256, 256, 256, 256, 256, 256, 256, 256, 256, 256, 256, 256, 256}};
constexpr signed char SST[10][16] = {{-1, -1, -1, 8, -1, -1, -1, -1, -1, 9, -1, -1, -1, -1, 10, -1}, {-1, -1, -1, 11, -1, -1, -1, -1, -1, 12, -1, -1, -1, -1, 13, -1}, {-1, -1, -1, 14, -1, -1, -1, -1, -1, 15, -1, -1, -1, -1, 16, -1}, {-1, -1, -1, 17, -1, -1, -1, -1, -1, 18, -1, -1, -1, -1, 19, -1}, {-1, -1, -1, 20, -1, -1, -1, -1, -1, 21, -1, -1, -1, -1, 22, -1}, {-1, -1, -1, 23, -1, -1, -1, -1, -1, 24, -1, -1, -1, -1, 25, -1}, {-1, -1, -1, 26, -1, -1, -1, -1, -1, 27, -1, -1, -1, -1, 28, -1}, {-1, -1, -1, 29, -1, -1, -1, -1, -1, 30, -1, -1, -1, -1, 31, -1}, {-1, -1, -1, -1, -1, -1, -1, -1, -1, -1, -1, -1, -1, -1, -1, -1}, {-1, -1, -1, -1, -1, -1, -1, -1, -1, -1, -1, -1, -1, -1, -1, -1}};
template <int Q, int Mi> DI void solo_gap(Ctx& c, unsigned& sh, unsigned& sw) {
  constexpr int s = Q >> 1, r = Q & 1, fx = Mi >> 2, iw = (fx & 1) ? 3 - (Mi & 3) : (Mi & 3);
  constexpr SoloQ q = SOLO[Q];
  mfma<AI(r, fx, 0, iw), false, XR(r) + 8 * fx, WR(s & 1) + 8 * iw>();
  if constexpr (q.r0 >= 0 && Mi < 8) ldsread<(q.r0 & 3) * GRP, Mi, XR(r ^ 1)>(q.r0 < 4 ? c.rdx : c.rdx2);
  if constexpr (q.r1 >= 0 && Mi >= 4 && Mi < 12) ldsread<(q.r1 & 3) * GRP, Mi - 4, WR((s + 1) & 1)>(q.r1 < 4 ? c.rdw : c.rdw2);
  if constexpr (r == 0) {
    if constexpr (Mi == 0) { sh = c.h2; sw = c.a2; asm volatile("" : "+s"(sh), "+s"(sw)); }
    if constexpr (q.dl >= 0) { if constexpr (Mi == 1) { pick0s(c); pick1(c); } if constexpr (Mi == 2) pick2(c); if constexpr (Mi == 3) pick3(c); }
    if constexpr (q.dw >= 0 && Mi >= 4 && Mi < 8) dma<q.dw * GRP, Mi - 4>(c, sw);
    if constexpr (q.dh >= 0 && Mi >= 8 && Mi < 12) dma<q.dh * GRP, Mi - 8>(c, sh);
    if constexpr (q.dl >= 0 && Mi >= 12) dma<q.dl * GRP, Mi - 12>(c, c.x2);
  } else if constexpr (q.dw >= 0 && Mi >= 12) dma<q.dw * GRP, Mi - 12>(c, c.a2);
  if constexpr (Q < 10) {
    if constexpr (SST[Q][Mi] >= 0) e_store<SST[Q][Mi], AUX_OUT>(c);
    e_ops<1, SOP[Q][Mi]>(c, std::make_integer_sequence<int, SOP[Q][Mi + 1] - SOP[Q][Mi]>{});
  }
  if constexpr (Mi == 15 && Q < 31) { __builtin_amdgcn_s_waitcnt(WAITCNT(q.wait, 0)); __builtin_amdgcn_s_barrier(); }
  GAP();
}
template <int Q, int... Ms> DI void solo_quad(Ctx& c, std::integer_sequence<int, Ms...>) { unsigned sh = 0, sw = 0; (solo_gap<Q, Ms>(c, sh, sw), ...); }
template <int... Qs> DI void solo(Ctx& c, std::integer_sequence<int, Qs...>) { (solo_quad<Qs>(c, std::make_integer_sequence<int, 16>{}), ...); }

struct Unit { bool isx; unsigned src, dst, flag; };
template <bool ORDERED> DI Unit unit_of(const Args& a, int u) {
  unsigned e = u >> 2;
  if (ORDERED) { const unsigned wd = RFL(a.order[u >> 3]); e = (u & 4) ? wd >> 16 : wd & 0xffffu; }
  const int sg = e >> 2, j = e & 3, w = (u >> 1) & 1, h = u & 1, ks = j * 8 + a.rank;
  Unit r; r.isx = sg < 32;
  r.src = RFL(r.isx ? (unsigned)((sg * 128 + w * 64) * KL + j * 128 + h * 64) : (unsigned)((j * 128 + h * 64) * N + col_of(sg - 32, w)));
  r.dst = RFL((unsigned)(sg * 32 + ks) * GRP + w * 8192 + h * 1024);
  r.flag = RFL((unsigned)((sg * 32 + ks) * 16 + (u & 3) * 4));
  return r;
}
DI void load_raw(unsigned (&raw)[16], const Unit& u, rsrc_t rx, rsrc_t rw, unsigned vx, unsigned vw) {
  if (u.isx) {
#pragma unroll
    for (int f = 0; f < 4; ++f) { const u32x4 v = __builtin_amdgcn_raw_buffer_load_b128(rx, vx, u.src + f * 16 * KL, 0); raw[4 * f] = v.x; raw[4 * f + 1] = v.y; raw[4 * f + 2] = v.z; raw[4 * f + 3] = v.w; }
  } else {
#pragma unroll
    for (int b = 0; b < 16; ++b) raw[b] = __builtin_amdgcn_raw_buffer_load_b32(rw, vw, u.src + b * N, 0);
  }
}
DI void cook(u32x4 (&out)[4], const unsigned (&raw)[16], bool isx) {
  if (isx) {
#pragma unroll
    for (int f = 0; f < 4; ++f) out[f] = (u32x4){raw[4 * f], raw[4 * f + 1], raw[4 * f + 2], raw[4 * f + 3]};
  } else {
#pragma unroll
    for (int b = 0; b < 4; ++b) {
      const unsigned r0 = raw[4 * b], r1 = raw[4 * b + 1], r2 = raw[4 * b + 2], r3 = raw[4 * b + 3];
      const unsigned t0 = __builtin_amdgcn_perm(r1, r0, 0x05010400u), t1 = __builtin_amdgcn_perm(r1, r0, 0x07030602u);
      const unsigned t2 = __builtin_amdgcn_perm(r3, r2, 0x05010400u), t3 = __builtin_amdgcn_perm(r3, r2, 0x07030602u);
      out[0][b] = __builtin_amdgcn_perm(t2, t0, 0x05040100u); out[1][b] = __builtin_amdgcn_perm(t2, t0, 0x07060302u);
      out[2][b] = __builtin_amdgcn_perm(t3, t1, 0x05040100u); out[3][b] = __builtin_amdgcn_perm(t3, t1, 0x07060302u);
    }
  }
}
DI unsigned read_call(char* myws) { return RFL(__hip_atomic_load((unsigned*)(myws + WS_DONE), __ATOMIC_RELAXED, __HIP_MEMORY_SCOPE_AGENT)) / GRID + 1; }
DI void pusher(const Args& a, char* myws, LAS char* lds, int pid, int wave, int lane) {
  const rsrc_t rx = mkrsrc(a.x, M * KL), rw = mkrsrc(a.w, KL * N);
  const unsigned vx = (lane & 15) * KL + (lane >> 4) * 16, vw = (lane >> 4) * 16 * N + (lane & 15) * 4, vd = wave * 2048 + lane * 16;
  unsigned raw[16]; u32x4 out[4];
  int un = pid + NPUSH * wave;
  Unit cur = unit_of<true>(a, un);
  load_raw(raw, cur, rx, rw, vx, vw);
  const unsigned call = read_call(myws);
  unsigned* uq = (unsigned*)(myws + WS_UNITQ);
  const rsrc_t ru = mkrsrc(uq, 4); const unsigned vu = lane == 0 ? 0u : OOB;
  rsrc_t rd[NR - 1];
  char* fl = nullptr;
#pragma unroll
  for (int i = 0; i < NR - 1; ++i) {
    const int d = (a.rank + 1 + (i + pid * 4 + wave) % 7) & 7;
    char* p = a.ws[RFL(d)];
    rd[i] = mkrsrc(p + WS_STAGE + (size_t)(call & 1) * STAGE_BYTES, STAGE_BYTES);
    fl = lane == i ? p + WS_FLAGS : fl;
  }
  LAS char* meta = lds + 32768;
  LAS unsigned* cnt = (LAS unsigned*)(lds + 32768 + 64);
  if (threadIdx.x < 4) cnt[threadIdx.x] = 0;
  unsigned fq[6] = {OOB, OOB, OOB, OOB, OOB, OOB};
  unsigned dsts[4];
  asm volatile("v_mov_b32 v252, %0\n\tv_mov_b32 v253, %1" :: "s"(call * 4096u + un + 4 * NPUSH), "s"(call * 4096u + un + 8 * NPUSH));
  auto done = [&](int slot, unsigned f) {
    unsigned old = 0;
    if (lane == 0) old = __hip_atomic_fetch_add(cnt + slot, 1u, __ATOMIC_RELAXED, __HIP_MEMORY_SCOPE_WORKGROUP);
    if ((RFL(old) & 3) == 3 && f != OOB && lane < NR - 1) __hip_atomic_store((unsigned*)(fl + f), call, __ATOMIC_RELAXED, __HIP_MEMORY_SCOPE_SYSTEM);
  };
  auto hand_over = [&](int set) {
    u32x4 ck[4]; cook(ck, raw, cur.isx);
#pragma unroll
    for (int i = 0; i < 4; ++i) *(LAS u32x4*)(lds + set * 16384 + wave * 4096 + i * 1024 + lane * 16) = ck[i];
    if (lane == 0) *(LAS u32x2*)(meta + set * 32 + wave * 8) = un < NU ? (u32x2){cur.dst, cur.flag} : (u32x2){OOB, OOB};
    { unsigned t; if (set) asm volatile("v_readfirstlane_b32 %0, v253" : "=s"(t)); else asm volatile("v_readfirstlane_b32 %0, v252" : "=s"(t)); un = t - call * 4096u; }
    cur = unit_of<true>(a, un < NU ? un : 0);
    load_raw(raw, cur, rx, rw, vx, vw);
  };
  __builtin_amdgcn_s_waitcnt(WAITCNT(0, 0));
  if (lane == 0) __hip_atomic_fetch_max(uq, call * 4096u + 12 * NPUSH, __ATOMIC_RELAXED, __HIP_MEMORY_SCOPE_AGENT);
  hand_over(0);
  int set = 0; bool more;
  do {
    __builtin_amdgcn_s_waitcnt(WAITCNT(63, 0));
    __builtin_amdgcn_s_barrier();
    const u32x4 m0 = *(LAS u32x4*)(meta + set * 32), m1 = *(LAS u32x4*)(meta + set * 32 + 16);
#pragma unroll
    for (int i = 0; i < 4; ++i) out[i] = *(LAS u32x4*)(lds + set * 16384 + i * 4096 + wave * 1024 + lane * 16);
    dsts[0] = RFL(m0.x); dsts[1] = RFL(m0.z); dsts[2] = RFL(m1.x); dsts[3] = RFL(m1.z);
    fq[0] = fq[4]; fq[1] = fq[5]; fq[2] = RFL(m0.y); fq[3] = RFL(m0.w); fq[4] = RFL(m1.y); fq[5] = RFL(m1.w);
    more = (dsts[0] & dsts[1] & dsts[2] & dsts[3]) != OOB;
#pragma unroll
    for (int i = 0; i < 4; ++i) {
      GAP();
      if (i == wave) {
        if (set) asm volatile("v_mov_b32 v253, 1\n\ts_nop 4\n\tbuffer_atomic_add v253, %0, %1, 0 offen sc0" :: "v"(vu), "s"(ru) : "memory");
        else asm volatile("v_mov_b32 v252, 1\n\ts_nop 4\n\tbuffer_atomic_add v252, %0, %1, 0 offen sc0" :: "v"(vu), "s"(ru) : "memory");
      }
#pragma unroll
      for (int s = 0; s < 7; ++s) __builtin_amdgcn_raw_buffer_store_b128(out[i], rd[s], vd, dsts[i], AUX_SYS);
      if (i == 1 || i == 2) __builtin_amdgcn_s_waitcnt(WAITCNT(14, 15)); else if (cur.isx) __builtin_amdgcn_s_waitcnt(WAITCNT(18, 15)); else __builtin_amdgcn_s_waitcnt(WAITCNT(30, 15));
      asm volatile("" ::: "memory");
      done((i + 2) & 3, fq[i]);
      if (i == 2) { GAP(); hand_over(set ^ 1); }
    }
    set ^= 1;
  } while (more);
}
constexpr int NTILE = 80;
DI unsigned local_tiling(const Args& a, char* myws, int gw, int lane) {
  constexpr int NW = NTILE * 4, NIT = (NU + NW - 1) / NW;
  if (gw >= NW) return read_call(myws);
  const rsrc_t rx = mkrsrc(a.x, M * KL), rw = mkrsrc(a.w, KL * N);
  const unsigned vx = (lane & 15) * KL + (lane >> 4) * 16, vw = (lane >> 4) * 16 * N + (lane & 15) * 4, vd = lane * 16;
  unsigned raw[2][16]; Unit un[2];
  static_assert(NW * NIT == NU, "whole rounds");
  un[0] = unit_of<false>(a, gw); un[1] = unit_of<false>(a, gw + NW);
  load_raw(raw[0], un[0], rx, rw, vx, vw); load_raw(raw[1], un[1], rx, rw, vx, vw);
  const unsigned call = read_call(myws);
  const rsrc_t rd = mkrsrc(myws + WS_STAGE + (size_t)(call & 1) * STAGE_BYTES, STAGE_BYTES);
  unsigned f = 0;
#pragma unroll
  for (int i = 0; i < NIT; ++i) {
    u32x4 out[4]; cook(out, raw[i & 1], un[i & 1].isx);
    const unsigned dst = un[i & 1].dst;
    f = lane == i ? un[i & 1].flag : f;
    GAP();
    if (i + 2 < NIT) { un[i & 1] = unit_of<false>(a, gw + (i + 2) * NW); load_raw(raw[i & 1], un[i & 1], rx, rw, vx, vw); }
#pragma unroll
    for (int s = 0; s < 4; ++s) __builtin_amdgcn_raw_buffer_store_b128(out[s], rd, vd, dst + s * 2048, AUX_SYS);
    GAP();
  }
  asm volatile("s_waitcnt vmcnt(0)" ::: "memory");
  if (lane < NIT) __hip_atomic_store((unsigned*)(myws + WS_FLAGS + f), call, __ATOMIC_RELAXED, __HIP_MEMORY_SCOPE_SYSTEM);
  return call;
}


DI void aim(Ctx& c, const Args& a, float scale, int strip) {
  c.rc = mkrsrc((char*)a.out + (size_t)(c.mt * 256 + (c.wave >> 1) * 64) * PITCH + col_of(strip < 0 ? 0 : strip, c.wave & 1) * 4, strip < 0 ? 0u : 192u * PITCH);
  c.sc = __builtin_bit_cast(float, RFL(*(volatile LAS unsigned*)(c.lds + LDS_DEAD) ? 0x7fc00000u : __builtin_bit_cast(unsigned, scale)));
}

__global__ void __launch_bounds__(256, 1) __attribute__((amdgpu_num_vgpr(64))) fused_gemm_gather(Args a) {
  extern __shared__ __attribute__((aligned(16))) char smem[];
  const int tid = threadIdx.x, lane = tid & 63, wave = RFL(tid >> 6);
  char* myws = a.ws[a.rank & 7];
  asm volatile("" ::: "a255", "v255");
  const bool is_push = blockIdx.x < NPUSH; const int pid = blockIdx.x, gid = blockIdx.x - NPUSH;
  if (is_push) {
    pusher(a, myws, (LAS char*)smem, pid, wave, lane);
  } else {
    Ctx c;
    c.lds = (LAS char*)smem;
    for (int i = tid; i < (int)(LDS_BYTES - LDS_RING) / 16; i += 256) *(volatile LAS u32x4*)(c.lds + LDS_RING + i * 16) = (u32x4){0u, 0u, 0u, 0u};
    __builtin_amdgcn_s_sleep(40);
    const unsigned call = RFL(local_tiling(a, myws, gid * 4 + wave, lane));
    const float scale = a.sx[0] * a.sw[0];
    const int bx = blockIdx.x, xq = bx & 3, half = (bx >> 2) & 1, idx = (bx >> 3) - 2;
    const int row = idx < 24 ? idx & 7 : (idx - 24) + (idx - 24 >= 2 * xq ? 2 : 0), slot = idx < 24 ? idx >> 3 : 3;
    c.mt = RFL(row * 2 + half); c.team = RFL(xq < 3 || slot < 3 ? 4 * xq + slot : 4 * (row >> 1) + 3);
    c.rs = mkrsrc(myws + WS_STAGE + (size_t)(call & 1) * STAGE_BYTES, STAGE_BYTES);
    c.rf = mkrsrc(myws + WS_FLAGS, FLAG_BYTES);
    c.call = call; c.tmo = (unsigned*)(myws + WS_TMO); c.wave = wave;
    c.vo = (lane >> 4) * 4 * PITCH + (lane & 15) * 16;
    const unsigned ldsb = (unsigned)(size_t)c.lds;
    c.rdx = ldsb + (wave >> 1) * 8192 + lane * 16; c.rdw = ldsb + (wave & 1) * 8192 + lane * 16; c.rdx2 = c.rdx + 65536; c.rdw2 = c.rdw + 65536;
    c.voff = wave * 4096 + lane * 16; c.m0w = wave * 4096;
    int h = (c.team >> 1) & 1;
    c.lead = RFL(lead_of(c.team));
    make_tables(c);
    c.notme = wave == 3 ? 0u : OOB; c.snapdst = RFL(LDS_SNAP + (wave == 3 ? 0 : 2048)); c.px = RFL(2 * c.mt * STRIP); asm volatile("" : "+s"(c.px), "+s"(c.snapdst)); c.seen = 0;
    c.snapv = flag_voff(c, h) | c.notme; c.wb = RFL((h * 16 + 16 - c.lead) & 31);
    open_half(c, h); c.quota = 11;
    __builtin_amdgcn_s_waitcnt(WAITCNT(63, 0));
    __builtin_amdgcn_s_barrier();
    pick0s(c); pick1(c); pick2(c); pick3(c);
    dma4<0 * GRP>(c, c.x2); dma4<1 * GRP>(c, c.a2); dma4<2 * GRP>(c, c.b2); dma4<3 * GRP>(c, c.x2 + STRIP);
    __builtin_amdgcn_s_barrier();
    pick0s(c); pick1(c); pick2(c); pick3(c);
    dma4<4 * GRP>(c, c.x2); dma4<5 * GRP>(c, c.b2); dma4<6 * GRP>(c, c.a2); dma4<7 * GRP>(c, c.x2 + STRIP);
    __builtin_amdgcn_s_barrier();
    pick0s(c); pick1(c); pick2(c); pick3(c);
    __builtin_amdgcn_s_waitcnt(WAITCNT(24, 15));
    __builtin_amdgcn_s_barrier();
    ldsread8<0 * GRP, XR(0)>(c.rdx); ldsread8<1 * GRP, WR(0)>(c.rdw);
    __builtin_amdgcn_s_waitcnt(WAITCNT(20, 0));
    __builtin_amdgcn_s_barrier();
    dma4<0 * GRP>(c, c.x2);
    aim(c, a, scale, -1);
    GAP();
#pragma unroll 1
    for (; h < 9; ++h) {
      if (h & 1) two_steps<2, 1>(c); else two_steps<2, 0>(c);
      int n = 6;
#pragma unroll 1
      do { tail(c); two_steps<0, 0>(c); asm volatile("" : "+s"(n)); } while (--n);
      tail(c);
      aim(c, a, scale, strip_of(h - 1, c.team));
      GAP();
      if (h & 1) two_steps<1, 0>(c); else two_steps<1, 1>(c);
    }
    solo(c, std::make_integer_sequence<int, 32>{});
    aim(c, a, scale, strip_of(8, c.team));
    asm volatile("s_nop 15");
    e_finish_(c, std::make_integer_sequence<int, 32>{});
  }
  if (tid == 0) __hip_atomic_fetch_add((unsigned*)(myws + WS_DONE), 1u, __ATOMIC_RELAXED, __HIP_MEMORY_SCOPE_AGENT);
}

}


static void build_order(Args& a) {
  float key[640]; unsigned short e[640];
  for (int i = 0; i < 640; ++i) {
    const int sg = i >> 2, j = i & 3, ks = j * 8 + a.rank; e[i] = (unsigned short)i;
    float f = sg / 32.0f; int t0 = 0;
    if (sg >= 32) {
      const int s = sg - 32, h0 = s < 8 ? 0 : 1 + (s - 8) / 15, team = s < 8 ? (s >> 1) * 4 + (s & 1) : (s - 8) % 15;
      t0 = 16 * h0 - lead_of(team); f = (team + 0.5f * (h0 & 1)) / 16.0f + 0.001f;
    }
    const int t = t0 + ((ks - t0) & 31);
    f += 0.06f * a.rank; f -= (int)f;
    key[i] = (t >> 3) + f;
  }
  for (int i = 1; i < 640; ++i) { const float k = key[i]; const unsigned short v = e[i]; int q = i - 1; while (q >= 0 && key[q] > k) { key[q + 1] = key[q]; e[q + 1] = e[q]; --q; } key[q + 1] = k; e[q + 1] = v; }
  for (int i = 0; i < 320; ++i) a.order[i] = e[2 * i] | ((unsigned)e[2 * i + 1] << 16);
}

extern "C" void kernel_launch(int rank, int nranks, ncclComm_t comm, hipStream_t stream,
                              void* const* d_in, const int* in_sizes, int n_in,
                              void* d_out, int out_size, const int* in_dtypes, int out_dtype,
                              void* const* d_out_all, void* const* d_ws_all, size_t ws_bytes) {
  if (nranks != NR || (unsigned)rank >= (unsigned)NR || n_in != 4 || in_sizes[0] != M * KL || in_sizes[1] != KL * N || out_size != M * N || ws_bytes < WS_NEED) {
    fprintf(stderr, "kernel_launch: unexpected problem shape\n");
    return;
  }
  Args a;
  a.x = (const unsigned char*)d_in[0]; a.w = (const unsigned char*)d_in[1]; a.sx = (const float*)d_in[2]; a.sw = (const float*)d_in[3];
  a.out = (float*)d_out; a.rank = rank;
  for (int p = 0; p < NR; ++p) a.ws[p] = (char*)d_ws_all[p];
  build_order(a);
  HIP_CHECK(hipFuncSetAttribute((const void*)fused_gemm_gather, hipFuncAttributeMaxDynamicSharedMemorySize, LDS_BYTES));
  hipLaunchKernelGGL(fused_gemm_gather, dim3(GRID), dim3(256), LDS_BYTES, stream, a);
  HIP_CHECK(hipGetLastError());
}
```
